```python
import numpy as np

kernel_name = "gpu_gemm_a2a_m4096_k4096_n16384_bf16_reluq8"


ABSMAX_RTOL = 0.02

COMPUTE_DTYPE = "bfloat16"
EPILOGUE = "relu_quant_int8"
IN_QUANT = None
OUT_QUANT = "int8"
IN_DTYPES = {"x": "bfloat16", "w_mat": "bfloat16"}
WORLD_SIZE = 8

def _to_bf16(x):
    u = np.ascontiguousarray(x, dtype=np.float32).view(np.uint32)
    return ((u + 0x7FFF + ((u >> 16) & 1)) & 0xFFFF0000).view(np.float32)

def setup_inputs():

    x = _to_bf16(np.random.randn(4096, 4096).astype(np.float32))
    w_mat = _to_bf16((np.random.randn(4096, 16384) / np.sqrt(4096)).astype(np.float32))
    return {"x": x, "w_mat": w_mat}

def _quant_dequant(y, amax):
    scale = amax / 127.0
    q = np.clip(np.round(y / scale), -127.0, 127.0)
    return (q * scale).astype(np.float32)
def _epilogue(y):
    y = np.maximum(y, 0.0)
    return _quant_dequant(y, np.abs(y).max())

def reference(x, w_mat):
    y = (x @ w_mat).astype(np.float32)
    return _epilogue(y)
```

```cpp
#include <hip/hip_runtime.h>
#include <rccl/rccl.h>
#include <cstdio>

#define HIP_CHECK(expr) do { hipError_t _e = (expr); if (_e != hipSuccess) { \
    fprintf(stderr, "HIP error %s at %s:%d\n", hipGetErrorString(_e), __FILE__, __LINE__); \
    return; } } while (0)

#define LAS __attribute__((address_space(3)))
typedef short v4s __attribute__((ext_vector_type(4)));
typedef short v8s __attribute__((ext_vector_type(8)));
typedef float f32x4 __attribute__((ext_vector_type(4)));
typedef unsigned u32x4 __attribute__((ext_vector_type(4)));
typedef unsigned long long u64;

namespace {
constexpr int NR = 8;
constexpr int KD = 4096, ND = 16384, ML = 512, NSH = ND / NR;
constexpr int BM = 256, BN = 128, BK = 64, NT = KD / BK;
constexpr int NWG = 256;
constexpr int LDS_A = 0, LDS_B = 98304, LDS_X = 147456, LDS_TOT = LDS_X + 64;
constexpr int WS_CNT = 0;
constexpr int WS_TMO = 4096;
constexpr int WS_LOC = 8192;
constexpr int WS_GLB = 16384;
constexpr int WS_DONE = 24576;
constexpr int WS_STAMP = 65536;
constexpr u64 DEADLINE = 50000000ull;

struct Params {
    const unsigned short* x; const unsigned short* w;
    char* out[NR]; char* ws[NR];
    int rank;
};
}

#define SB0 __builtin_amdgcn_sched_barrier(0)
#define WAITV(n) asm volatile("s_waitcnt vmcnt(" #n ")" ::: "memory")
#define WAITL0 do { asm volatile("" ::: "memory"); __builtin_amdgcn_s_waitcnt(0xC07F); asm volatile("" ::: "memory"); } while (0)
#define BAR do { asm volatile("" ::: "memory"); __builtin_amdgcn_s_barrier(); asm volatile("" ::: "memory"); } while (0)
#define STAMP(i) do { } while (0)

__device__ __forceinline__ v8s cat(v4s a, v4s b) { return __builtin_shufflevector(a, b, 0, 1, 2, 3, 4, 5, 6, 7); }
__device__ __forceinline__ unsigned wave_umax(unsigned v) {
#pragma unroll
    for (int o = 32; o >= 1; o >>= 1) v = max(v, (unsigned)__shfl_xor((int)v, o));
    return v;
}
__device__ __forceinline__ bool give_up(unsigned* tmo, unsigned call, u64 t0) {
    if (__hip_atomic_load(tmo, __ATOMIC_RELAXED, __HIP_MEMORY_SCOPE_AGENT) == call || __builtin_amdgcn_s_memrealtime() - t0 > DEADLINE) {
        __hip_atomic_store(tmo, call, __ATOMIC_RELAXED, __HIP_MEMORY_SCOPE_AGENT);
        return true;
    }
    return false;
}

__global__ __launch_bounds__(512, 2) void fused_gemm_a2a(Params p) {
    __shared__ __attribute__((aligned(1024))) char smem[LDS_TOT];
    LAS char* lds = (LAS char*)smem;
    volatile LAS unsigned* xs = (volatile LAS unsigned*)(lds + LDS_X);
    const int tid = threadIdx.x, lane = tid & 63;
    const int wid = __builtin_amdgcn_readfirstlane(tid >> 6);
    const int b = blockIdx.x, dsel = b & 7, idx = b >> 3;
    const int rb = idx & 1, cb = idx >> 1;
    const int dest = (p.rank + dsel) & 7;
    const int m0 = rb * BM, n0 = dest * NSH + cb * BN;
    char* myws = p.ws[p.rank];
    unsigned* tmo = (unsigned*)(myws + WS_TMO);
    STAMP(0);

    if (wid >= 4) {
        const int lw = wid - 4;
        auto rx = __builtin_amdgcn_make_buffer_rsrc((void*)p.x, 0, ML * KD * 2, 0x00020000);
        auto rw = __builtin_amdgcn_make_buffer_rsrc((void*)p.w, 0, KD * ND * 2, 0x00020000);
        const int voA = (m0 + lw * 8 + (lane >> 3)) * (KD * 2) + (((lane & 7) ^ ((lw & 1) * 4 + (lane >> 4))) * 16);
        const int voB = (lw * 4 + (lane >> 4)) * (ND * 2) + n0 * 2 + (((lane & 15) ^ (((lw & 2) << 2) | (lane >> 4))) * 16);
        auto issue = [&](int t, int so) {
#pragma unroll
            for (int j = 0; j < 4; ++j)
                __builtin_amdgcn_raw_ptr_buffer_load_lds(rw, (LAS void*)(lds + LDS_B + so + lw * 1024 + j * 4096), 16, voB,
                                                         t * (BK * ND * 2) + j * (16 * ND * 2), 0, 0);
#pragma unroll
            for (int j = 0; j < 8; ++j)
                __builtin_amdgcn_raw_ptr_buffer_load_lds(rx, (LAS void*)(lds + LDS_A + (j >> 2) * 49152 + so + lw * 1024 + (j & 3) * 4096), 16, voA,
                                                         t * (BK * 2) + j * (32 * KD * 2), 0, 0);
        };
        issue(0, 0); issue(1, 16384); issue(2, 32768);
        WAITV(24); BAR;
        int so = 0;
        for (int t = 0; t < NT - 3; ++t) {
            WAITV(12); BAR;
            issue(t + 3, so);
            so = (so == 32768) ? 0 : so + 16384;
        }
        WAITV(12); BAR;
        WAITV(0); BAR;
        BAR;
        BAR;
        const unsigned call = xs[4];
        const int par = call & 1;
        if (lw == 0) {
            const unsigned mx = max(max(xs[0], xs[1]), max(xs[2], xs[3]));
            if (lane == 0) __hip_atomic_store((u64*)(myws + WS_LOC) + par * NWG + b, ((u64)call << 32) | mx, __ATOMIC_RELAXED, __HIP_MEMORY_SCOPE_AGENT);
            STAMP(1);
            const u64* g = (const u64*)(myws + WS_GLB) + (par * 8 + (idx & 7)) * 16 + (lane & 7);
            const u64 t0 = __builtin_amdgcn_s_memrealtime();
            unsigned bad = 0; u64 v;
            for (unsigned it = 1;; ++it) {
                v = __hip_atomic_load(g, __ATOMIC_RELAXED, __HIP_MEMORY_SCOPE_SYSTEM);
                if (__all((unsigned)(v >> 32) == call)) break;
                if ((it & 63u) == 0u && __builtin_amdgcn_readfirstlane((int)give_up(tmo, call, t0))) { bad = 1; break; }
                __builtin_amdgcn_s_sleep(1);
            }
            unsigned am = (unsigned)v;
            am = max(am, (unsigned)__shfl_xor((int)am, 1)); am = max(am, (unsigned)__shfl_xor((int)am, 2)); am = max(am, (unsigned)__shfl_xor((int)am, 4));
            if (lane == 0) { xs[5] = am; xs[6] = bad; }
            WAITL0;
            STAMP(2);
        } else if (lw == 1 && b == 0) {
            const u64* tb = (const u64*)(myws + WS_LOC) + par * NWG + lane;
            const u64 t0 = __builtin_amdgcn_s_memrealtime();
            unsigned mx; bool dead = false;
            for (unsigned it = 1;; ++it) {
                bool ok = true; mx = 0;
#pragma unroll
                for (int i = 0; i < 4; ++i) { const u64 v = __hip_atomic_load(tb + i * 64, __ATOMIC_RELAXED, __HIP_MEMORY_SCOPE_AGENT); ok &= (unsigned)(v >> 32) == call; mx = max(mx, (unsigned)v); }
                if (__all(ok)) break;
                if ((it & 63u) == 0u && __builtin_amdgcn_readfirstlane((int)give_up(tmo, call, t0))) { dead = true; break; }
                __builtin_amdgcn_s_sleep(1);
            }
            mx = wave_umax(mx);
            STAMP(1);
            if (!dead) {
                char* base = p.ws[0];
#pragma unroll
                for (int r = 1; r < NR; ++r) base = ((lane & 7) == r) ? p.ws[r] : base;
                __hip_atomic_store((u64*)(base + WS_GLB) + (par * 8 + (lane >> 3)) * 16 + p.rank, ((u64)call << 32) | mx, __ATOMIC_RELAXED, __HIP_MEMORY_SCOPE_SYSTEM);
            }
        }
        BAR;
        if (b == 0) {
            const int src = lw * 2 + (lane >> 5);
            const unsigned* f = (const unsigned*)(myws + WS_DONE) + src * 128 + (lane & 31) * 4;
            const u64 t0 = __builtin_amdgcn_s_memrealtime();
            for (unsigned it = 1;; ++it) {
                bool ok = true;
#pragma unroll
                for (int i = 0; i < 4; ++i) ok &= (__hip_atomic_load(f + i, __ATOMIC_RELAXED, __HIP_MEMORY_SCOPE_SYSTEM) >= call);
                if (__all(ok || src == p.rank)) break;
                if ((it & 63u) == 0u && __builtin_amdgcn_readfirstlane((int)give_up(tmo, call, t0))) {
                    if (lane == 0) p.out[p.rank][(size_t)src * ML * NSH] = (char)-128;
                    if (lane == 32) p.out[p.rank][(size_t)src * ML * NSH] = (char)-128;
                    break;
                }
                __builtin_amdgcn_s_sleep(1);
            }
            STAMP(3);
        }
        return;
    }
    unsigned call = 0;
    if (tid == 0) call = __hip_atomic_fetch_add((unsigned*)(myws + WS_CNT) + b, 1u, __ATOMIC_RELAXED, __HIP_MEMORY_SCOPE_AGENT) + 1u;
    const int wm = wid >> 1, wn = wid & 1;
    const int fr = lane & 15, fq = lane >> 4;
    const int aA0 = LDS_A + wm * 49152 + fr * 128 + ((fq ^ (fr >> 1)) * 16);
    const int aA1 = aA0 ^ 64;
    int aB[4];
    {
        const int q = fr >> 2, pp = lane & 3;
        const int ch = (wn * 8 + 2 * pp) ^ (((fq & 1) << 3) | q);
        const int a0 = LDS_B + (8 * fq + q) * 256 + ch * 16 + (pp & 1) * 8;
#pragma unroll
        for (int f = 0; f < 4; ++f) aB[f] = a0 ^ (8 * f);
    }
    f32x4 acc[8][4];
#pragma unroll
    for (int m = 0; m < 8; ++m)
#pragma unroll
        for (int f = 0; f < 4; ++f) acc[m][f] = (f32x4){0.f, 0.f, 0.f, 0.f};
    v8s A[4][2];
    v4s B0[4][2][2], B1[4][2][2];
#define RA(m, ks, so) A[(m) & 3][ks] = *(LAS v8s*)(lds + ((ks) ? aA1 : aA0) + (so) + (m) * 2048)
#define RB(Bv, f, ks, t, so) Bv[f][ks][t] = __builtin_amdgcn_ds_read_tr16_b64_v4i16((LAS v4s*)(lds + aB[f] + (so) + (ks) * 8192 + (t) * 1024))
#define MF(m, f, ks, Bv) asm volatile("v_mfma_f32_16x16x32_bf16 %0, %1, %2, %0" : "+v"(acc[m][f]) : "v"(cat(Bv[f][ks][0], Bv[f][ks][1])), "v"(A[(m) & 3][ks]) : "memory")
#define MF2(m, f, Bv) asm volatile("v_mfma_f32_16x16x32_bf16 %0, %1, %2, %0\n\tv_mfma_f32_16x16x32_bf16 %0, %3, %4, %0" : "+v"(acc[m][f]) : "v"(cat(Bv[f][0][0], Bv[f][0][1])), "v"(A[(m) & 3][0]), "v"(cat(Bv[f][1][0], Bv[f][1][1])), "v"(A[(m) & 3][1]) : "memory")
    BAR;
    STAMP(1);
    WAITL0; SB0;
#pragma unroll
    for (int f = 0; f < 4; ++f) { RB(B0, f, 0, 0, 0); RB(B0, f, 0, 1, 0); RB(B0, f, 1, 0, 0); RB(B0, f, 1, 1, 0); }
#pragma unroll
    for (int m = 0; m < 3; ++m) { RA(m, 0, 0); RA(m, 1, 0); }
    int so = 0;
#define GRP_A(m, Bc, ma, soa) \
        MF(m, 0, 0, Bc); RA(ma, 0, soa); SB0; MF(m, 0, 1, Bc); RA(ma, 1, soa); SB0; MF2(m, 1, Bc); MF2(m, 2, Bc); MF2(m, 3, Bc); SB0;
#define KTILE(Bc, Bn) { \
        const int sn = (so == 32768) ? 0 : so + 16384; \
        SB0; \
        GRP_A(0, Bc, 3, so) GRP_A(1, Bc, 4, so) GRP_A(2, Bc, 5, so) GRP_A(3, Bc, 6, so) GRP_A(4, Bc, 7, so) \
        WAITL0; BAR; SB0; \
        RB(Bn, 0, 0, 0, sn); SB0; \
        MF(5, 0, 0, Bc); RB(Bn, 0, 0, 1, sn); SB0; MF(5, 0, 1, Bc); RA(0, 0, sn); SB0; MF(5, 1, 0, Bc); RB(Bn, 0, 1, 0, sn); SB0; MF(5, 1, 1, Bc); RB(Bn, 0, 1, 1, sn); SB0; \
        MF(5, 2, 0, Bc); RA(0, 1, sn); SB0; MF(5, 2, 1, Bc); RB(Bn, 1, 0, 0, sn); SB0; MF(5, 3, 0, Bc); RB(Bn, 1, 0, 1, sn); SB0; MF(5, 3, 1, Bc); RB(Bn, 1, 1, 0, sn); SB0; \
        MF(6, 0, 0, Bc); RB(Bn, 1, 1, 1, sn); SB0; MF(6, 0, 1, Bc); RB(Bn, 2, 0, 0, sn); SB0; MF(6, 1, 0, Bc); RB(Bn, 2, 0, 1, sn); SB0; MF(6, 1, 1, Bc); RB(Bn, 2, 1, 0, sn); SB0; \
        MF(6, 2, 0, Bc); RB(Bn, 2, 1, 1, sn); SB0; MF(6, 2, 1, Bc); RB(Bn, 3, 0, 0, sn); SB0; MF(6, 3, 0, Bc); RB(Bn, 3, 0, 1, sn); SB0; MF(6, 3, 1, Bc); RB(Bn, 3, 1, 0, sn); SB0; \
        MF(7, 0, 0, Bc); RB(Bn, 3, 1, 1, sn); SB0; MF(7, 0, 1, Bc); RA(1, 0, sn); SB0; MF(7, 1, 0, Bc); RA(1, 1, sn); SB0; MF(7, 1, 1, Bc); RA(2, 0, sn); SB0; \
        MF(7, 2, 0, Bc); RA(2, 1, sn); SB0; MF(7, 2, 1, Bc); MF2(7, 3, Bc); SB0; \
        so = sn; }
    for (int t = 0; t < NT; t += 2) {
        KTILE(B0, B1)
        KTILE(B1, B0)
    }
    asm volatile("s_nop 7\n\ts_nop 7" ::: "memory");
    SB0;
    STAMP(2);
    float mx = 0.f;
#pragma unroll
    for (int m = 0; m < 8; ++m)
#pragma unroll
        for (int f = 0; f < 4; ++f) mx = fmaxf(fmaxf(mx, fmaxf(acc[m][f][0], acc[m][f][1])), fmaxf(acc[m][f][2], acc[m][f][3]));
    const unsigned wmx = wave_umax(__float_as_uint(mx));
    if (lane == 0) { xs[wid] = wmx; if (wid == 0) xs[4] = call; }
    WAITL0;
    BAR;
#pragma unroll
    for (int m = 0; m < 8; ++m)
#pragma unroll
        for (int f = 0; f < 4; ++f)
#pragma unroll
            for (int r = 0; r < 4; ++r) acc[m][f][r] = fmaxf(acc[m][f][r], 0.f);
    auto ro = __builtin_amdgcn_make_buffer_rsrc((void*)p.out[dest], 0, NR * ML * NSH, 0x00020000);
    const int voff = (p.rank * ML + m0 + wm * 128 + fr) * NSH + cb * BN + wn * 64 + fq * 16;
    const bool odd = fq & 1;
    BAR;
    STAMP(3);
    call = xs[4];
    const float amax = __uint_as_float(xs[5]);
    const bool bad = xs[6] != 0u;
    const float inv = amax > 0.f ? 127.0f / amax : 0.f;
#pragma unroll
    for (int m = 0; m < 8; ++m) {
        unsigned d[4];
#pragma unroll
        for (int f = 0; f < 4; ++f) {
            const unsigned q0 = __float_as_uint(fmaf(acc[m][f][0], inv, 12582912.0f)), q1 = __float_as_uint(fmaf(acc[m][f][1], inv, 12582912.0f));
            const unsigned q2 = __float_as_uint(fmaf(acc[m][f][2], inv, 12582912.0f)), q3 = __float_as_uint(fmaf(acc[m][f][3], inv, 12582912.0f));
            const unsigned lo = __builtin_amdgcn_perm(q1, q0, 0x0c0c0400u), hi = __builtin_amdgcn_perm(q3, q2, 0x04000c0cu);
            d[f] = lo | hi;
        }
        u32x4 o;
        o.x = odd ? d[1] : d[0]; o.y = odd ? d[0] : d[1]; o.z = odd ? d[3] : d[2]; o.w = odd ? d[2] : d[3];
        if (bad) o = (u32x4){0x80808080u, 0x80808080u, 0x80808080u, 0x80808080u};
        __builtin_amdgcn_raw_buffer_store_b128(o, ro, voff + m * 16 * NSH, 0, 17);
    }
    STAMP(4);
    WAITV(0);
    if (lane == 0 && dsel != 0)
        __hip_atomic_store((unsigned*)(p.ws[dest] + WS_DONE) + p.rank * 128 + idx * 4 + wid, call, __ATOMIC_RELAXED, __HIP_MEMORY_SCOPE_SYSTEM);
    STAMP(5);
}

extern "C" void kernel_launch(int rank, int nranks, ncclComm_t comm, hipStream_t stream,
                              void* const* d_in, const int* in_sizes, int n_in,
                              void* d_out, int out_size, const int* in_dtypes, int out_dtype,
                              void* const* d_out_all, void* const* d_ws_all, size_t ws_bytes) {
    if (nranks != NR || n_in < 2 || in_sizes[0] != ML * KD || in_sizes[1] != KD * ND || out_size != NR * ML * NSH || ws_bytes < (1u << 20) || (unsigned)rank >= (unsigned)NR) {
        fprintf(stderr, "kernel_launch: unexpected problem shape\n");
        return;
    }
    Params p;
    p.x = (const unsigned short*)d_in[0];
    p.w = (const unsigned short*)d_in[1];
    for (int r = 0; r < NR; ++r) { p.out[r] = (char*)d_out_all[r]; p.ws[r] = (char*)d_ws_all[r]; }
    p.rank = rank;
    hipLaunchKernelGGL(fused_gemm_a2a, dim3(NWG), dim3(512), 0, stream, p);
    HIP_CHECK(hipGetLastError());
}
```

```cpp
#include <hip/hip_runtime.h>
#include <rccl/rccl.h>
#include <cstdio>

#define HIP_CHECK(expr) do { hipError_t _e = (expr); if (_e != hipSuccess) { \
    fprintf(stderr, "HIP error %s at %s:%d\n", hipGetErrorString(_e), __FILE__, __LINE__); \
    return; } } while (0)

#define LAS __attribute__((address_space(3)))
typedef short v4s __attribute__((ext_vector_type(4)));
typedef short v8s __attribute__((ext_vector_type(8)));
typedef float f32x4 __attribute__((ext_vector_type(4)));
typedef unsigned u32x4 __attribute__((ext_vector_type(4)));
typedef unsigned long long u64;

#define KROT 8
#define WIN 0

namespace {
constexpr int NR = 8;
constexpr int KD = 4096, ND = 16384, ML = 512, NSH = ND / NR;
constexpr int BM = 256, BN = 128, BK = 64, NT = KD / BK;
constexpr int NWG = 256;
constexpr int LDS_A = 0, LDS_B = 98304, LDS_X = 147456, LDS_LUT = LDS_X + 64, LDS_TOT = LDS_LUT + 256;
constexpr int NREP = 32;
constexpr int WS_CNT = 0;
constexpr int WS_TMO = 4096;
constexpr int WS_LOC = 8192;
constexpr int WS_GLB = 16384;
constexpr int WS_STAMP = 65536;
constexpr int WS_MSK = 1 << 20;
constexpr int WS_DAT = 3 << 20;
constexpr int MSK_PAR = 1 << 20, DAT_BYTES = 8 << 20;
constexpr size_t WS_NEED = (size_t)WS_DAT + (size_t)DAT_BYTES;
constexpr u64 DEADLINE = 50000000ull;
constexpr int OOB = 0x40000000;

struct Params {
    const unsigned short* x; const unsigned short* w;
    char* out; char* ws[NR];
    int rank;
};
}

#define SB0 __builtin_amdgcn_sched_barrier(0)
#define WAITV(n) asm volatile("s_waitcnt vmcnt(%0)" :: "n"(n) : "memory")
#define WAITL0 do { asm volatile("" ::: "memory"); __builtin_amdgcn_s_waitcnt(0xC07F); asm volatile("" ::: "memory"); } while (0)
#define BAR do { asm volatile("" ::: "memory"); __builtin_amdgcn_s_barrier(); asm volatile("" ::: "memory"); } while (0)
#define STAMP(i) do { } while (0)

__device__ __forceinline__ v8s cat(v4s a, v4s b) { return __builtin_shufflevector(a, b, 0, 1, 2, 3, 4, 5, 6, 7); }
__device__ __forceinline__ int imax3(int a, float b, float c) { return max(max(a, __float_as_int(b)), __float_as_int(c)); }
__device__ __forceinline__ float relu(float a) { return __int_as_float(max(__float_as_int(a), 0)); }
__device__ __forceinline__ unsigned wave_umax(unsigned v) {
    v = max(v, (unsigned)__builtin_amdgcn_update_dpp(0, (int)v, 0xB1, 0xf, 0xf, true));
    v = max(v, (unsigned)__builtin_amdgcn_update_dpp(0, (int)v, 0x4E, 0xf, 0xf, true));
    v = max(v, (unsigned)__builtin_amdgcn_update_dpp(0, (int)v, 0x141, 0xf, 0xf, true));
    v = max(v, (unsigned)__builtin_amdgcn_update_dpp(0, (int)v, 0x140, 0xf, 0xf, true));
    const unsigned a = __builtin_amdgcn_readlane(v, 0), b = __builtin_amdgcn_readlane(v, 16), c = __builtin_amdgcn_readlane(v, 32), d = __builtin_amdgcn_readlane(v, 48);
    return max(max(a, b), max(c, d));
}
__device__ __forceinline__ bool give_up(unsigned* tmo, unsigned call, u64 t0) {
    if (__hip_atomic_load(tmo, __ATOMIC_RELAXED, __HIP_MEMORY_SCOPE_AGENT) == call || __builtin_amdgcn_s_memrealtime() - t0 > DEADLINE) {
        __hip_atomic_store(tmo, call, __ATOMIC_RELAXED, __HIP_MEMORY_SCOPE_AGENT);
        return true;
    }
    return false;
}
__device__ __forceinline__ unsigned quant4(float a, float b, float c, float d, float inv) {
    const unsigned q0 = __float_as_uint(fmaf(a, inv, 12582912.0f)), q1 = __float_as_uint(fmaf(b, inv, 12582912.0f));
    const unsigned q2 = __float_as_uint(fmaf(c, inv, 12582912.0f)), q3 = __float_as_uint(fmaf(d, inv, 12582912.0f));
    return __builtin_amdgcn_perm(q1, q0, 0x0c0c0400u) | __builtin_amdgcn_perm(q3, q2, 0x04000c0cu);
}
__device__ __forceinline__ int chunk_off(int n, int c) {
    const bool act = n > 16 * c;
    const u64 bal = __ballot(act);
    const int rank = __builtin_amdgcn_mbcnt_hi((unsigned)(bal >> 32), __builtin_amdgcn_mbcnt_lo((unsigned)bal, 0));
    return act ? c * 1024 + rank * 16 : OOB;
}

__device__ __forceinline__ unsigned tag_word(unsigned call, int i) {
    const unsigned t = ((call % 65535u) + 1u) >> (4 * i);
    return ((t & 1u) << 7) | ((t & 2u) << 14) | ((t & 4u) << 21) | ((t & 8u) << 28);
}

__global__ __launch_bounds__(512, 2) void fused_gemm_a2a(Params p) {
    __shared__ __attribute__((aligned(1024))) char smem[LDS_TOT];
    LAS char* lds = (LAS char*)smem;
    volatile LAS unsigned* xs = (volatile LAS unsigned*)(lds + LDS_X);
    const int tid = threadIdx.x, lane = tid & 63;
    const int wid = __builtin_amdgcn_readfirstlane(tid >> 6);
    const int b = blockIdx.x, dsel = b & 7, idx = b >> 3;
    const int rb = idx & 1, cb = idx >> 1;
    const int m0 = rb * BM;
    char* myws = p.ws[0];
    unsigned* tmo = (unsigned*)(myws + WS_TMO);
    unsigned call;
#define GET_CALL() do { call = *(const volatile __attribute__((address_space(4))) unsigned*)(myws + WS_CNT) + 1u; asm volatile("" : "+s"(call)); } while (0)
    const int fr = lane & 15, fq = lane >> 4;
    STAMP(0);

    if (wid >= 4) {
        const int lw = wid - 4;
        {
            const int n0 = ((p.rank + dsel) & 7) * NSH + cb * BN;
            auto rx = __builtin_amdgcn_make_buffer_rsrc((void*)p.x, 0, ML * KD * 2, 0x00020000);
            auto rw = __builtin_amdgcn_make_buffer_rsrc((void*)p.w, 0, KD * ND * 2, 0x00020000);
            const int voA = (m0 + lw * 8 + (lane >> 3)) * (KD * 2) + (((lane & 7) ^ ((lw & 1) * 4 + (lane >> 4))) * 16);
            const int voB = (lw * 4 + (lane >> 4)) * (ND * 2) + n0 * 2 + (((lane & 15) ^ (((lw & 2) << 2) | (lane >> 4))) * 16);
            auto issue = [&](int t, int so) {
                const int kt = (t + dsel * KROT) & (NT - 1);
#pragma unroll
                for (int j = 0; j < 4; ++j)
                    __builtin_amdgcn_raw_ptr_buffer_load_lds(rw, (LAS void*)(lds + LDS_B + so + lw * 1024 + j * 4096), 16, voB,
                                                             kt * (BK * ND * 2) + j * (16 * ND * 2), 0, 0);
#pragma unroll
                for (int j = 0; j < 8; ++j)
                    __builtin_amdgcn_raw_ptr_buffer_load_lds(rx, (LAS void*)(lds + LDS_A + (j >> 2) * 49152 + so + lw * 1024 + (j & 3) * 4096), 16, voA,
                                                             kt * (BK * 2) + j * (32 * KD * 2), 0, 0);
            };
            issue(0, 0); issue(1, 16384); issue(2, 32768);
            GET_CALL();
            WAITV(24); BAR;
            int so = 0;
            for (int t = 0; t < NT - 3; ++t) {
                WAITV(12); BAR;
                issue(t + 3, so);
                so = (so == 32768) ? 0 : so + 16384;
            }
            WAITV(12); BAR;
            {
                auto rm = __builtin_amdgcn_make_buffer_rsrc((void*)(myws + WS_MSK), 0, 2 * MSK_PAR, 0x00020000);
                WAITV(0);
                __builtin_amdgcn_raw_buffer_store_b128((u32x4){0u, 0u, 0u, 0u}, rm, lane * 16, ((call & 1) ^ 1) * MSK_PAR + (b * 4 + lw) * 1024, 17);
            }
            BAR;
            BAR;
            WAITV(0);
        }
        const int par = call & 1;
        if (lw == 2) {
            const int nib = lane & 15, al = lane >> 4;
            unsigned sel = 0;
#pragma unroll
            for (int r = 0; r < 4; ++r) sel |= (((nib >> r) & 1) ? (unsigned)(al + __builtin_popcount(nib & ((1 << r) - 1))) : 0x0cu) << (8 * r);
            *(LAS unsigned*)(lds + LDS_LUT + lane * 4) = sel;
            if (lane < 4) xs[4 + lane] = 0u;
            WAITL0;
        }
        BAR;
        if (lw == 0 && lane == 0) {
            const unsigned mx = max(max(xs[0], xs[1]), max(xs[2], xs[3]));
            __hip_atomic_store((u64*)(myws + WS_LOC) + par * NWG + b, ((u64)call << 32) | mx, __ATOMIC_RELAXED, __HIP_MEMORY_SCOPE_AGENT);
        }
        STAMP(1);
        if (lw == 1 && b == 0) {
            char* base = p.ws[0];
#pragma unroll
            for (int r = 1; r < NR; ++r) base = ((lane & 7) == r) ? p.ws[r] : base;
            u64* dst = (u64*)(base + WS_GLB) + (par * NREP + (lane >> 3)) * 16 + p.rank;
            asm volatile("" : "+v"(dst));
            const u64* tb = (const u64*)(myws + WS_LOC) + par * NWG + lane;
            const u64 t0 = __builtin_amdgcn_s_memrealtime();
            unsigned mx; bool dead = false;
            for (unsigned it = 1;; ++it) {
                bool ok = true; mx = 0;
#pragma unroll
                for (int i = 0; i < 4; ++i) { const u64 v = __hip_atomic_load(tb + i * 64, __ATOMIC_RELAXED, __HIP_MEMORY_SCOPE_AGENT); ok &= (unsigned)(v >> 32) == call; mx = max(mx, (unsigned)v); }
                if (__all(ok)) break;
                if ((it & 63u) == 0u && __builtin_amdgcn_readfirstlane((int)give_up(tmo, call, t0))) { dead = true; break; }
                __builtin_amdgcn_s_sleep(1);
            }
            mx = wave_umax(mx);
            if (!dead) {
#pragma unroll
                for (int i = 0; i < NREP / 8; ++i) __hip_atomic_store(dst + i * 128, ((u64)call << 32) | mx, __ATOMIC_RELAXED, __HIP_MEMORY_SCOPE_SYSTEM);
            }
            if (lane == 0) __hip_atomic_store((unsigned*)(myws + WS_CNT), call, __ATOMIC_RELAXED, __HIP_MEMORY_SCOPE_AGENT);
            STAMP(6);
        }
        const int slot = b * 4 + lw;
        const u64 t0 = __builtin_amdgcn_s_memrealtime();
        bool ok = true;
        u32x4 kept;
        {
            auto rm = __builtin_amdgcn_make_buffer_rsrc((void*)(myws + WS_MSK), 0, 2 * MSK_PAR, 0x00020000);
            for (unsigned it = 1;; ++it) {
                kept = __builtin_amdgcn_raw_buffer_load_b128(rm, lane * 16, par * MSK_PAR + slot * 1024, 17);
                if (__all((int)(kept[0] & kept[1] & kept[2] & kept[3]) < 0)) break;
                if ((it & 63u) == 0u && __builtin_amdgcn_readfirstlane((int)give_up(tmo, call, t0))) { ok = false; break; }
                __builtin_amdgcn_s_sleep(16);
                asm volatile("" ::: "memory");
            }
        }
        STAMP(2);
        const int sbase = lw * 32768 + lane * 4;
        int wa[32]; unsigned sel[32]; int n = 0;
#pragma unroll
        for (int w = 0; w < 4; ++w) {
#pragma unroll
            for (int j = 0; j < 8; ++j) {
                const int sh = 4 * (j ^ (fq & 1));
                const int pos = n + __builtin_popcount(__builtin_amdgcn_ubfe(kept[w], 0, sh));
                const int nib = __builtin_amdgcn_ubfe(kept[w], sh, 4);
                wa[w * 8 + j] = sbase + (pos >> 2) * 256;
                sel[w * 8 + j] = *(LAS unsigned*)(lds + LDS_LUT + (((pos & 3) << 4) | nib) * 4);
            }
            n += __builtin_popcount(kept[w]);
        }
        int coff[8];
#pragma unroll
        for (int c = 0; c < 8; ++c) coff[c] = chunk_off(n, c);
#pragma unroll
        for (int i = 0; i < 32; ++i) { asm volatile("" : "+v"(wa[i])); asm volatile("" : "+v"(sel[i])); }
        const int src = (p.rank - dsel) & 7;
        char* o = p.out + (size_t)(src * ML + m0 + (lw >> 1) * 128 + fr) * NSH + cb * BN + (lw & 1) * 64 + fq * 16;
        asm volatile("" : "+v"(o));
        const unsigned tg0 = tag_word(call, 0), tg1 = tag_word(call, 1), tg2 = tag_word(call, 2), tg3 = tag_word(call, 3);
        while (xs[4 + lw] == 0u) __builtin_amdgcn_s_sleep(1);
        asm volatile("" ::: "memory");
        STAMP(3);
        {
            auto rd = __builtin_amdgcn_make_buffer_rsrc((void*)(myws + WS_DAT), 0, DAT_BYTES, 0x00020000);
            for (unsigned it = 1; ok; ++it) {
                const u32x4 v = __builtin_amdgcn_raw_buffer_load_b128(rd, lane == 0 ? 0 : OOB, slot * 8192, 17);
                if (__builtin_amdgcn_readfirstlane((((v[0] ^ tg0) | (v[1] ^ tg1) | (v[2] ^ tg2) | (v[3] ^ tg3)) & 0x80808080u) == 0u)) break;
                if ((it & 63u) == 0u && __builtin_amdgcn_readfirstlane((int)give_up(tmo, call, t0))) ok = false;
                __builtin_amdgcn_s_sleep(8);
                asm volatile("" ::: "memory");
            }
            STAMP(7);
            for (unsigned it = 1; ok; ++it) {
                u32x4 v[8];
#pragma unroll
                for (int c = 0; c < 8; ++c) v[c] = __builtin_amdgcn_raw_buffer_load_b128(rd, coff[c], slot * 8192, 17);
                int left = OOB;
#pragma unroll
                for (int c = 0; c < 8; ++c) {
                    const bool got = coff[c] != OOB && ((((v[c][0] ^ tg0) | (v[c][1] ^ tg1) | (v[c][2] ^ tg2) | (v[c][3] ^ tg3)) & 0x80808080u) == 0u);
                    if (got) {
#pragma unroll
                        for (int i = 0; i < 4; ++i) *(LAS unsigned*)(lds + sbase + (c * 4 + i) * 256) = v[c][i] & 0x7f7f7f7fu;
                        coff[c] = OOB;
                    }
                    left = min(left, coff[c]);
                }
                if (__all(left == OOB)) break;
                if ((it & 63u) == 0u && __builtin_amdgcn_readfirstlane((int)give_up(tmo, call, t0))) ok = false;
                __builtin_amdgcn_s_sleep(2);
                asm volatile("" ::: "memory");
            }
        }
        STAMP(4);
#pragma unroll
        for (int m = 0; m < 8; ++m) {
            u32x4 ov;
#pragma unroll
            for (int j = 0; j < 4; ++j) {
                const unsigned lo = *(LAS unsigned*)(lds + wa[m * 4 + j]), hi = *(LAS unsigned*)(lds + wa[m * 4 + j] + 256);
                ov[j] = __builtin_amdgcn_perm(hi, lo, sel[m * 4 + j]);
            }
            if (!ok) ov = (u32x4){0x80808080u, 0x80808080u, 0x80808080u, 0x80808080u};
            *(u32x4*)(o + (size_t)m * 16 * NSH) = ov;
        }
        STAMP(5);
        return;
    }
    const int wm = wid >> 1, wn = wid & 1;
    const int aA0 = LDS_A + wm * 49152 + fr * 128 + ((fq ^ (fr >> 1)) * 16);
    const int aA1 = aA0 ^ 64;
    int aB[4];
    {
        const int q = fr >> 2, pp = lane & 3;
        const int ch = (wn * 8 + 2 * pp) ^ (((fq & 1) << 3) | q);
        const int a0 = LDS_B + (8 * fq + q) * 256 + ch * 16 + (pp & 1) * 8;
#pragma unroll
        for (int f = 0; f < 4; ++f) aB[f] = a0 ^ (8 * f);
    }
    f32x4 acc[8][4];
#pragma unroll
    for (int m = 0; m < 8; ++m)
#pragma unroll
        for (int f = 0; f < 4; ++f) { acc[m][f] = (f32x4){0.f, 0.f, 0.f, 0.f}; asm volatile("" : "+v"(acc[m][f])); }
    v8s A[4][2];
    v4s B0[4][2][2], B1[4][2][2];
#define RA(m, ks, so) A[(m) & 3][ks] = *(LAS v8s*)(lds + ((ks) ? aA1 : aA0) + (so) + (m) * 2048)
#define RB(Bv, f, ks, t, so) Bv[f][ks][t] = __builtin_amdgcn_ds_read_tr16_b64_v4i16((LAS v4s*)(lds + aB[f] + (so) + (ks) * 8192 + (t) * 1024))
#define MF(m, f, ks, Bv) asm volatile("v_mfma_f32_16x16x32_bf16 %0, %1, %2, %0" : "+v"(acc[m][f]) : "v"(cat(Bv[f][ks][0], Bv[f][ks][1])), "v"(A[(m) & 3][ks]) : "memory")
#define MF2(m, f, Bv) asm volatile("v_mfma_f32_16x16x32_bf16 %0, %1, %2, %0\n\tv_mfma_f32_16x16x32_bf16 %0, %3, %4, %0" : "+v"(acc[m][f]) : "v"(cat(Bv[f][0][0], Bv[f][0][1])), "v"(A[(m) & 3][0]), "v"(cat(Bv[f][1][0], Bv[f][1][1])), "v"(A[(m) & 3][1]) : "memory")
    GET_CALL();
    BAR;
    STAMP(1);
    WAITL0; SB0;
#pragma unroll
    for (int f = 0; f < 4; ++f) { RB(B0, f, 0, 0, 0); RB(B0, f, 0, 1, 0); RB(B0, f, 1, 0, 0); RB(B0, f, 1, 1, 0); }
#pragma unroll
    for (int m = 0; m < 3; ++m) { RA(m, 0, 0); RA(m, 1, 0); }
    int so = 0;
#define GRP_A(m, Bc, ma, soa) \
        MF(m, 0, 0, Bc); RA(ma, 0, soa); SB0; MF(m, 0, 1, Bc); RA(ma, 1, soa); SB0; MF2(m, 1, Bc); MF2(m, 2, Bc); MF2(m, 3, Bc); SB0;
#define KTILE(Bc, Bn) { \
        const int sn = (so == 32768) ? 0 : so + 16384; \
        SB0; \
        GRP_A(0, Bc, 3, so) GRP_A(1, Bc, 4, so) GRP_A(2, Bc, 5, so) GRP_A(3, Bc, 6, so) GRP_A(4, Bc, 7, so) \
        WAITL0; BAR; SB0; \
        RB(Bn, 0, 0, 0, sn); SB0; \
        MF(5, 0, 0, Bc); RB(Bn, 0, 0, 1, sn); SB0; MF(5, 0, 1, Bc); RA(0, 0, sn); SB0; MF(5, 1, 0, Bc); RB(Bn, 0, 1, 0, sn); SB0; MF(5, 1, 1, Bc); RB(Bn, 0, 1, 1, sn); SB0; \
        MF(5, 2, 0, Bc); RA(0, 1, sn); SB0; MF(5, 2, 1, Bc); RB(Bn, 1, 0, 0, sn); SB0; MF(5, 3, 0, Bc); RB(Bn, 1, 0, 1, sn); SB0; MF(5, 3, 1, Bc); RB(Bn, 1, 1, 0, sn); SB0; \
        MF(6, 0, 0, Bc); RB(Bn, 1, 1, 1, sn); SB0; MF(6, 0, 1, Bc); RB(Bn, 2, 0, 0, sn); SB0; MF(6, 1, 0, Bc); RB(Bn, 2, 0, 1, sn); SB0; MF(6, 1, 1, Bc); RB(Bn, 2, 1, 0, sn); SB0; \
        MF(6, 2, 0, Bc); RB(Bn, 2, 1, 1, sn); SB0; MF(6, 2, 1, Bc); RB(Bn, 3, 0, 0, sn); SB0; MF(6, 3, 0, Bc); RB(Bn, 3, 0, 1, sn); SB0; MF(6, 3, 1, Bc); RB(Bn, 3, 1, 0, sn); SB0; \
        MF(7, 0, 0, Bc); RB(Bn, 3, 1, 1, sn); SB0; MF(7, 0, 1, Bc); RA(1, 0, sn); SB0; MF(7, 1, 0, Bc); RA(1, 1, sn); SB0; MF(7, 1, 1, Bc); RA(2, 0, sn); SB0; \
        MF(7, 2, 0, Bc); RA(2, 1, sn); SB0; MF(7, 2, 1, Bc); MF2(7, 3, Bc); SB0; \
        so = sn; }
    for (int t = 0; t < NT; t += 2) {
        KTILE(B0, B1)
        KTILE(B1, B0)
    }
    asm volatile("s_nop 7\n\ts_nop 7" ::: "memory");
    SB0;
    STAMP(2);
    {
        int mx = 0;
#pragma unroll
        for (int m = 0; m < 8; ++m)
#pragma unroll
            for (int f = 0; f < 4; ++f) mx = imax3(imax3(mx, acc[m][f][0], acc[m][f][1]), acc[m][f][2], acc[m][f][3]);
        const unsigned wmx = wave_umax((unsigned)mx);
        if (lane == 0) xs[wid] = wmx;
        WAITL0;
        BAR;
    }
    const int par = call & 1;
    const int slot = b * 4 + wid;
    char* pws = p.ws[dsel];
#pragma unroll
    for (int w = 0; w < 4; ++w) acc[2 * w + 1][3][3] = relu(acc[2 * w + 1][3][3]);
    u32x4 kept;
#pragma unroll
    for (int w = 0; w < 4; ++w) {
        unsigned sg = 0;
#pragma unroll
        for (int i = 31; i >= 0; --i) { const int v = w * 32 + i; sg = __builtin_amdgcn_alignbit(sg, __float_as_uint(acc[v >> 4][(v >> 2) & 3][v & 3]), 31); }
        kept[w] = ~sg;
    }
    {
        auto rm = __builtin_amdgcn_make_buffer_rsrc((void*)(pws + WS_MSK), 0, 2 * MSK_PAR, 0x00020000);
        __builtin_amdgcn_raw_buffer_store_b128(kept, rm, lane * 16, par * MSK_PAR + slot * 1024, 17);
    }
    SB0;
    {
        const int a0 = wid * 32768 + lane * 4;
        int a = a0;
#pragma unroll
        for (int v = 0; v < 128; ++v) {
            const float y = acc[v >> 4][(v >> 2) & 3][v & 3];
            *(LAS float*)(lds + a + (127 - v) * 256) = y;
            int t;
            if (v < 127) asm("v_lshrrev_b32 %1, 31, %2\n\tv_lshl_add_u32 %0, %1, 8, %0" : "+v"(a), "=&v"(t) : "v"(y));
        }
        asm volatile("" ::: "memory");
        SB0;
#pragma unroll
        for (int v = 0; v < 128; ++v) acc[v >> 4][(v >> 2) & 3][v & 3] = *(LAS float*)(lds + a0 + (127 - v) * 256);
    }
    const int n = __builtin_popcount(kept[0]) + __builtin_popcount(kept[1]) + __builtin_popcount(kept[2]) + __builtin_popcount(kept[3]);
    const int nch = (int)((wave_umax((unsigned)n) + 15u) >> 4);
    int coff[8];
#pragma unroll
    for (int c = 0; c < 8; ++c) coff[c] = chunk_off(n, c);
#pragma unroll
    for (int m = 0; m < 8; ++m) {
        asm volatile("" : "+v"(coff[m]));
#pragma unroll
        for (int f = 0; f < 4; ++f) asm volatile("" : "+v"(acc[m][f]));
    }
    WAITL0;
    if (lane == 0) xs[4 + wid] = 1u;
    auto rd = __builtin_amdgcn_make_buffer_rsrc((void*)(pws + WS_DAT), 0, DAT_BYTES, 0x00020000);
    STAMP(6);
    unsigned tg[4];
#pragma unroll
    for (int i = 0; i < 4; ++i) { tg[i] = tag_word(call, i); asm volatile("" : "+s"(tg[i])); }
    u64 gv;
    {
        const u64* g = (const u64*)(myws + WS_GLB) + (par * NREP + idx) * 16 + (lane & 7);
        const u64 t0 = __builtin_amdgcn_s_memrealtime();
        for (unsigned it = 1;; ++it) {
            gv = __hip_atomic_load(g, __ATOMIC_RELAXED, __HIP_MEMORY_SCOPE_SYSTEM);
            if (__all((unsigned)(gv >> 32) == call)) break;
            if ((it & 63u) == 0u && __builtin_amdgcn_readfirstlane((int)give_up(tmo, call, t0))) return;
            __builtin_amdgcn_s_sleep(1);
        }
    }
    STAMP(3);
    const float amax = __uint_as_float(wave_umax((unsigned)gv));
    const float inv = amax > 0.f ? 127.0f / amax : 0.f;
#define SEND(c) do { u32x4 ov; \
        _Pragma("unroll") for (int f = 0; f < 4; ++f) ov[f] = (quant4(acc[c][f][0], acc[c][f][1], acc[c][f][2], acc[c][f][3], inv) & 0x7f7f7f7fu) | tg[f]; \
        if ((c) >= 1 && (c) <= 3) WAITV(WIN); \
        __builtin_amdgcn_raw_buffer_store_b128(ov, rd, coff[c], slot * 8192, 17); } while (0)
    SEND(0);
    if (nch > 7) SEND(7);
    if (nch > 6) SEND(6);
    if (nch > 5) SEND(5);
    if (nch > 4) SEND(4);
    if (nch > 3) SEND(3);
    if (nch > 2) SEND(2);
    if (nch > 1) SEND(1);
    STAMP(4);
}

extern "C" void kernel_launch(int rank, int nranks, ncclComm_t comm, hipStream_t stream,
                              void* const* d_in, const int* in_sizes, int n_in,
                              void* d_out, int out_size, const int* in_dtypes, int out_dtype,
                              void* const* d_out_all, void* const* d_ws_all, size_t ws_bytes) {
    if (nranks != NR || n_in < 2 || in_sizes[0] != ML * KD || in_sizes[1] != KD * ND || out_size != NR * ML * NSH || ws_bytes < WS_NEED || (unsigned)rank >= (unsigned)NR) {
        fprintf(stderr, "kernel_launch: unexpected problem shape\n");
        return;
    }
    Params p;
    p.x = (const unsigned short*)d_in[0];
    p.w = (const unsigned short*)d_in[1];
    p.out = (char*)d_out;
    for (int d = 0; d < NR; ++d) p.ws[d] = (char*)d_ws_all[(rank + d) & 7];
    p.rank = rank;
    hipLaunchKernelGGL(fused_gemm_a2a, dim3(NWG), dim3(512), 0, stream, p);
    HIP_CHECK(hipGetLastError());
}
```

```cpp
#include <hip/hip_runtime.h>
#include <rccl/rccl.h>
#include <cstdio>

#define HIP_CHECK(expr) do { hipError_t _e = (expr); if (_e != hipSuccess) { \
    fprintf(stderr, "HIP error %s at %s:%d\n", hipGetErrorString(_e), __FILE__, __LINE__); \
    return; } } while (0)

#define LAS __attribute__((address_space(3)))
typedef short v4s __attribute__((ext_vector_type(4)));
typedef short v8s __attribute__((ext_vector_type(8)));
typedef float f32x4 __attribute__((ext_vector_type(4)));
typedef unsigned u32x4 __attribute__((ext_vector_type(4)));
typedef unsigned long long u64;

#define KROT 8
#define WNT 0
#define OAUX 16
#define WIN 0

namespace {
constexpr int NR = 8;
constexpr int KD = 4096, ND = 16384, ML = 512, NSH = ND / NR;
constexpr int BM = 256, BN = 128, BK = 64, NT = KD / BK;
constexpr int NWG = 256;
constexpr int LDS_B = 0, LDS_A = 65536, LDS_X = 131072, LDS_LUT = LDS_X + 64, LDS_TOT = 163840;
constexpr int NREP = 32;
constexpr int WS_CNT = 0;
constexpr int WS_TMO = 4096;
constexpr int WS_LOC = 8192;
constexpr int WS_GLB = 16384;
constexpr int WS_STAMP = 65536;
constexpr int WS_MSK = 1 << 20;
constexpr int WS_DAT = 3 << 20;
constexpr int MSK_PAR = 1 << 20, DAT_BYTES = 8 << 20;
constexpr size_t WS_NEED = (size_t)WS_DAT + (size_t)DAT_BYTES;
constexpr u64 DEADLINE = 50000000ull;
constexpr int OOB = 0x40000000;

struct Params {
    const unsigned short* x; const unsigned short* w;
    int rank, pad;
    char* out; char* ws[NR];
};
}

#define SB0 __builtin_amdgcn_sched_barrier(0)
#define WAITV(n) asm volatile("s_waitcnt vmcnt(%0)" :: "n"(n) : "memory")
#define WAITL0 do { asm volatile("" ::: "memory"); __builtin_amdgcn_s_waitcnt(0xC07F); asm volatile("" ::: "memory"); } while (0)
#define BAR do { asm volatile("" ::: "memory"); __builtin_amdgcn_s_barrier(); asm volatile("" ::: "memory"); } while (0)
#define STAMP(i) do { } while (0)

__device__ __forceinline__ v8s cat(v4s a, v4s b) { return __builtin_shufflevector(a, b, 0, 1, 2, 3, 4, 5, 6, 7); }
__device__ __forceinline__ int imax3(int a, float b, float c) { return max(max(a, __float_as_int(b)), __float_as_int(c)); }
__device__ __forceinline__ float relu(float a) { return __int_as_float(max(__float_as_int(a), 0)); }
__device__ __forceinline__ unsigned wave_umax(unsigned v) {
    v = max(v, (unsigned)__builtin_amdgcn_update_dpp(0, (int)v, 0xB1, 0xf, 0xf, true));
    v = max(v, (unsigned)__builtin_amdgcn_update_dpp(0, (int)v, 0x4E, 0xf, 0xf, true));
    v = max(v, (unsigned)__builtin_amdgcn_update_dpp(0, (int)v, 0x141, 0xf, 0xf, true));
    v = max(v, (unsigned)__builtin_amdgcn_update_dpp(0, (int)v, 0x140, 0xf, 0xf, true));
    const unsigned a = __builtin_amdgcn_readlane(v, 0), b = __builtin_amdgcn_readlane(v, 16), c = __builtin_amdgcn_readlane(v, 32), d = __builtin_amdgcn_readlane(v, 48);
    return max(max(a, b), max(c, d));
}
__device__ __forceinline__ bool give_up(unsigned* tmo, unsigned call, u64 t0) {
    if (__hip_atomic_load(tmo, __ATOMIC_RELAXED, __HIP_MEMORY_SCOPE_AGENT) == call || __builtin_amdgcn_s_memrealtime() - t0 > DEADLINE) {
        __hip_atomic_store(tmo, call, __ATOMIC_RELAXED, __HIP_MEMORY_SCOPE_AGENT);
        return true;
    }
    return false;
}
__device__ __forceinline__ unsigned quant4(float a, float b, float c, float d, float inv, unsigned slo, unsigned shi) {
    const unsigned q0 = __float_as_uint(fmaf(a, inv, 12582912.0f)), q1 = __float_as_uint(fmaf(b, inv, 12582912.0f));
    const unsigned q2 = __float_as_uint(fmaf(c, inv, 12582912.0f)), q3 = __float_as_uint(fmaf(d, inv, 12582912.0f));
    return __builtin_amdgcn_perm(q1, q0, slo) | __builtin_amdgcn_perm(q3, q2, shi);
}
__device__ __forceinline__ int chunk_off(int total, int fq, int c) {
    const bool act = total > 16 * (4 * c + fq);
    const u64 bal = __ballot(act);
    const int rank = __builtin_amdgcn_mbcnt_hi((unsigned)(bal >> 32), __builtin_amdgcn_mbcnt_lo((unsigned)bal, 0));
    return act ? c * 1024 + rank * 16 : OOB;
}
__device__ __forceinline__ void pool_scan(int n, int fr, int fq, int& base, int& total) {
    const int n0 = __shfl(n, fr), n1 = __shfl(n, fr + 16), n2 = __shfl(n, fr + 32), n3 = __shfl(n, fr + 48);
    base = (fq > 0 ? n0 : 0) + (fq > 1 ? n1 : 0) + (fq > 2 ? n2 : 0);
    total = (n0 + n1) + (n2 + n3);
}
__device__ __forceinline__ unsigned tag_word(unsigned call, int i) {
    const unsigned t = ((call % 65535u) + 1u) >> (4 * i);
    return ((t & 1u) << 7) | ((t & 2u) << 14) | ((t & 4u) << 21) | ((t & 8u) << 28);
}

__global__ __launch_bounds__(512, 2) void fused_gemm_a2a(Params p) {
    __shared__ __attribute__((aligned(1024))) char smem[LDS_TOT];
    LAS char* lds = (LAS char*)smem;
    volatile LAS unsigned* xs = (volatile LAS unsigned*)(lds + LDS_X);
    const int tid = threadIdx.x, lane = tid & 63;
    const int wid = __builtin_amdgcn_readfirstlane(tid >> 6);
    const int b = blockIdx.x, dsel = b & 7, idx = b >> 3;
    const int rb = idx & 1, cb = idx >> 1;
    const int m0 = rb * BM;
    const int n0 = ((p.rank + dsel) & 7) * NSH + cb * BN;
    auto rx = __builtin_amdgcn_make_buffer_rsrc((void*)p.x, 0, ML * KD * 2, 0x00020000);
    auto rw = __builtin_amdgcn_make_buffer_rsrc((void*)p.w, 0, KD * ND * 2, 0x00020000);
    const int voA = (m0 + wid * 8 + (lane >> 3)) * (KD * 2) + (((lane & 7) ^ ((wid & 1) * 4 + (lane >> 4))) * 16);
    const int voB = (wid * 4 + (lane >> 4)) * (ND * 2) + n0 * 2 + (((lane & 15) ^ (((wid & 2) << 2) | (lane >> 4))) * 16);
    {
        const int kt = (dsel * KROT) & (NT - 1);
#pragma unroll
        for (int j = 0; j < 2; ++j)
            __builtin_amdgcn_raw_ptr_buffer_load_lds(rw, (LAS void*)(lds + LDS_B + wid * 1024 + j * 8192), 16, voB, kt * (BK * ND * 2) + j * (32 * ND * 2), 0, WNT);
#pragma unroll
        for (int j = 0; j < 4; ++j)
            __builtin_amdgcn_raw_ptr_buffer_load_lds(rx, (LAS void*)(lds + LDS_A + (j >> 1) * 49152 + wid * 1024 + (j & 1) * 8192), 16, voA, kt * (BK * 2) + j * (64 * KD * 2), 0, 0);
    }
    SB0;
    char* myws = p.ws[0];
    unsigned* tmo = (unsigned*)(myws + WS_TMO);
    unsigned call;
#define GET_CALL() do { call = *(const volatile __attribute__((address_space(4))) unsigned*)(myws + WS_CNT) + 1u; asm volatile("" : "+s"(call)); } while (0)
    const int fr = lane & 15, fq = lane >> 4;
    STAMP(0);

    if (__builtin_expect(wid >= 4, 1)) {
        const int lw = wid - 4;
        {
            const int vA = voA - 32 * (KD * 2), vB = voB - 16 * (ND * 2);
            const int ksw = (WNT ? cb : (rb ^ cb)) & 1;
            auto kt_of = [&](int t) { return ((t < 2 ? t : t ^ ksw) + dsel * KROT) & (NT - 1); };
            auto issueW = [&](int t) {
                const int so = t < NT ? kt_of(t) * (BK * ND * 2) : OOB;
#pragma unroll
                for (int j = 0; j < 4; ++j)
                    __builtin_amdgcn_raw_ptr_buffer_load_lds(rw, (LAS void*)(lds + LDS_B + (t & 3) * 16384 + lw * 1024 + j * 4096), 16, vB, so + j * (16 * ND * 2), 0, WNT);
            };
            auto issueX = [&](int t, int so) {
                const int kt = kt_of(t);
#pragma unroll
                for (int j = 0; j < 8; ++j)
                    __builtin_amdgcn_raw_ptr_buffer_load_lds(rx, (LAS void*)(lds + LDS_A + (j >> 2) * 49152 + so + lw * 1024 + (j & 3) * 4096), 16, vA, kt * (BK * 2) + j * (32 * KD * 2), 0, 0);
            };
            issueW(1); issueW(2); issueX(1, 16384); issueW(3); issueX(2, 32768);
            GET_CALL();
            WAITV(28); BAR;
            WAITV(24); BAR;
            int so = 0;
            for (int t = 0; t < NT - 3; ++t) {
                WAITV(12); BAR;
                issueW(t + 4); issueX(t + 3, so);
                so = (so == 32768) ? 0 : so + 16384;
            }
            WAITV(12); BAR;
            {
                auto rm = __builtin_amdgcn_make_buffer_rsrc((void*)(myws + WS_MSK), 0, 2 * MSK_PAR, 0x00020000);
                WAITV(0);
                __builtin_amdgcn_raw_buffer_store_b128((u32x4){0u, 0u, 0u, 0u}, rm, lane * 16, ((call & 1) ^ 1) * MSK_PAR + (b * 4 + lw) * 1024, 17);
            }
            BAR;
            BAR;
            WAITV(0);
        }
        const int par = call & 1;
        if (lw == 2) {
            const int nib = lane & 15, al = lane >> 4;
            unsigned sel = 0;
#pragma unroll
            for (int r = 0; r < 4; ++r) sel |= (((nib >> r) & 1) ? (unsigned)(al + __builtin_popcount(nib & ((1 << r) - 1))) : 0x0cu) << (8 * r);
            *(LAS unsigned*)(lds + LDS_LUT + lane * 4) = sel;
            if (lane < 4) xs[4 + lane] = 0u;
            WAITL0;
        }
        BAR;
        if (lw == 0 && lane == 0) {
            const unsigned mx = max(max(xs[0], xs[1]), max(xs[2], xs[3]));
            __hip_atomic_store((u64*)(myws + WS_LOC) + par * NWG + b, ((u64)call << 32) | mx, __ATOMIC_RELAXED, __HIP_MEMORY_SCOPE_AGENT);
        }
        STAMP(1);
        if (lw == 1 && b == 0) {
            char* base = p.ws[0];
#pragma unroll
            for (int r = 1; r < NR; ++r) base = ((lane & 7) == r) ? p.ws[r] : base;
            u64* dst = (u64*)(base + WS_GLB) + (par * NREP + (lane >> 3)) * 16 + p.rank;
            asm volatile("" : "+v"(dst));
            const u64* tb = (const u64*)(myws + WS_LOC) + par * NWG + lane;
            const u64 t0 = __builtin_amdgcn_s_memrealtime();
            unsigned mx; bool dead = false;
            for (unsigned it = 1;; ++it) {
                bool ok = true; mx = 0;
#pragma unroll
                for (int i = 0; i < 4; ++i) { const u64 v = __hip_atomic_load(tb + i * 64, __ATOMIC_RELAXED, __HIP_MEMORY_SCOPE_AGENT); ok &= (unsigned)(v >> 32) == call; mx = max(mx, (unsigned)v); }
                if (__all(ok)) break;
                if ((it & 63u) == 0u && __builtin_amdgcn_readfirstlane((int)give_up(tmo, call, t0))) { dead = true; break; }
                __builtin_amdgcn_s_sleep(1);
            }
            mx = wave_umax(mx);
            if (!dead) {
#pragma unroll
                for (int i = 0; i < NREP / 8; ++i) __hip_atomic_store(dst + i * 128, ((u64)call << 32) | mx, __ATOMIC_RELAXED, __HIP_MEMORY_SCOPE_SYSTEM);
            }
            if (lane == 0) __hip_atomic_store((unsigned*)(myws + WS_CNT), call, __ATOMIC_RELAXED, __HIP_MEMORY_SCOPE_AGENT);
            STAMP(6);
        }
        const int slot = b * 4 + lw;
        const u64 t0 = __builtin_amdgcn_s_memrealtime();
        bool ok = true;
        u32x4 kept;
        {
            auto rm = __builtin_amdgcn_make_buffer_rsrc((void*)(myws + WS_MSK), 0, 2 * MSK_PAR, 0x00020000);
            for (unsigned it = 1;; ++it) {
                kept = __builtin_amdgcn_raw_buffer_load_b128(rm, lane * 16, par * MSK_PAR + slot * 1024, 17);
                if (__all((int)kept[3] < 0)) break;
                if ((it & 63u) == 0u && __builtin_amdgcn_readfirstlane((int)give_up(tmo, call, t0))) { ok = false; break; }
                __builtin_amdgcn_s_sleep(16);
                asm volatile("" ::: "memory");
            }
        }
        STAMP(2);
        const int sbase = lw * 32768 + fr * 4;
        int wa[32]; unsigned sel[32]; int n, total;
        pool_scan(__builtin_popcount(kept[0]) + __builtin_popcount(kept[1]) + __builtin_popcount(kept[2]) + __builtin_popcount(kept[3]), fr, fq, n, total);
#pragma unroll
        for (int w = 0; w < 4; ++w) {
#pragma unroll
            for (int j = 0; j < 8; ++j) {
                const int sh = 4 * (j ^ (fq & 1));
                const int pos = n + __builtin_popcount(__builtin_amdgcn_ubfe(kept[w], 0, sh));
                const int nib = __builtin_amdgcn_ubfe(kept[w], sh, 4);
                wa[w * 8 + j] = sbase + (pos >> 2) * 64;
                sel[w * 8 + j] = *(LAS unsigned*)(lds + LDS_LUT + (((pos & 3) << 4) | nib) * 4);
            }
            n += __builtin_popcount(kept[w]);
        }
        int coff[8];
#pragma unroll
        for (int c = 0; c < 8; ++c) coff[c] = chunk_off(total, fq, c);
#pragma unroll
        for (int i = 0; i < 32; ++i) { asm volatile("" : "+v"(wa[i])); asm volatile("" : "+v"(sel[i])); }
        const int src = (p.rank - dsel) & 7;
        int o = (src * ML + m0 + (lw >> 1) * 128 + fr) * NSH + cb * BN + (lw & 1) * 64 + fq * 16;
        asm volatile("" : "+v"(o));
        auto ro = __builtin_amdgcn_make_buffer_rsrc((void*)p.out, 0, NR * ML * NSH, 0x00020000);
        const unsigned tg0 = tag_word(call, 0), tg1 = tag_word(call, 1), tg2 = tag_word(call, 2), tg3 = tag_word(call, 3);
        while (xs[4 + lw] == 0u) __builtin_amdgcn_s_sleep(1);
        asm volatile("" ::: "memory");
        STAMP(3);
        {
            auto rd = __builtin_amdgcn_make_buffer_rsrc((void*)(myws + WS_DAT), 0, DAT_BYTES, 0x00020000);
            for (unsigned it = 1; ok; ++it) {
                const u32x4 v = __builtin_amdgcn_raw_buffer_load_b128(rd, lane == 0 ? 0 : OOB, slot * 8192, 17);
                if (__builtin_amdgcn_readfirstlane((((v[0] ^ tg0) | (v[1] ^ tg1) | (v[2] ^ tg2) | (v[3] ^ tg3)) & 0x80808080u) == 0u)) break;
                if ((it & 63u) == 0u && __builtin_amdgcn_readfirstlane((int)give_up(tmo, call, t0))) ok = false;
                __builtin_amdgcn_s_sleep(8);
                asm volatile("" ::: "memory");
            }
            STAMP(7);
            for (unsigned it = 1; ok; ++it) {
                u32x4 v[8];
#pragma unroll
                for (int c = 0; c < 8; ++c) v[c] = __builtin_amdgcn_raw_buffer_load_b128(rd, coff[c], slot * 8192, 17);
                int left = OOB;
#pragma unroll
                for (int c = 0; c < 8; ++c) {
                    const bool got = coff[c] != OOB && ((((v[c][0] ^ tg0) | (v[c][1] ^ tg1) | (v[c][2] ^ tg2) | (v[c][3] ^ tg3)) & 0x80808080u) == 0u);
                    if (got) {
#pragma unroll
                        for (int i = 0; i < 4; ++i) *(LAS unsigned*)(lds + sbase + fq * 256 + (c * 16 + i) * 64) = v[c][i] & 0x7f7f7f7fu;
                        coff[c] = OOB;
                    }
                    left = min(left, coff[c]);
                }
                if (__all(left == OOB)) break;
                if ((it & 63u) == 0u && __builtin_amdgcn_readfirstlane((int)give_up(tmo, call, t0))) ok = false;
                __builtin_amdgcn_s_sleep(2);
                asm volatile("" ::: "memory");
            }
        }
        STAMP(4);
#pragma unroll
        for (int m = 0; m < 8; ++m) {
            u32x4 ov;
#pragma unroll
            for (int j = 0; j < 4; ++j) {
                const unsigned lo = *(LAS unsigned*)(lds + wa[m * 4 + j]), hi = *(LAS unsigned*)(lds + wa[m * 4 + j] + 64);
                ov[j] = __builtin_amdgcn_perm(hi, lo, sel[m * 4 + j]);
            }
            if (!ok) ov = (u32x4){0x80808080u, 0x80808080u, 0x80808080u, 0x80808080u};
            __builtin_amdgcn_raw_buffer_store_b128(ov, ro, o, m * 16 * NSH, OAUX);
        }
        STAMP(5);
        return;
    }
    const int wm = wid >> 1, wn = wid & 1;
    const int aA0 = LDS_A + wm * 49152 + fr * 128 + ((fq ^ (fr >> 1)) * 16);
    const int aA1 = aA0 ^ 64;
    int aB[4];
    {
        const int q = fr >> 2, pp = lane & 3;
        const int ch = (wn * 8 + 2 * pp) ^ (((fq & 1) << 3) | q);
        const int a0 = LDS_B + (8 * fq + q) * 256 + ch * 16 + (pp & 1) * 8;
#pragma unroll
        for (int f = 0; f < 4; ++f) aB[f] = a0 ^ (8 * f);
    }
    f32x4 acc[8][4];
#pragma unroll
    for (int m = 0; m < 8; ++m)
#pragma unroll
        for (int f = 0; f < 4; ++f) { acc[m][f] = (f32x4){0.f, 0.f, 0.f, 0.f}; asm volatile("" : "+v"(acc[m][f])); }
    v8s A[4][2];
    v4s B0[4][2][2], B1[4][2][2];
#define RA(m, ks, so) A[(m) & 3][ks] = *(LAS v8s*)(lds + ((ks) ? aA1 : aA0) + (so) + (m) * 2048)
#define RB(Bv, i, sb) Bv[(i) >> 2][((i) >> 1) & 1][(i) & 1] = __builtin_amdgcn_ds_read_tr16_b64_v4i16((LAS v4s*)(lds + aB[(i) >> 2] + (sb) * 16384 + (((i) >> 1) & 1) * 8192 + ((i) & 1) * 1024))
#define MF(m, f, ks, Bv) asm volatile("v_mfma_f32_16x16x32_bf16 %0, %1, %2, %0" : "+v"(acc[m][f]) : "v"(cat(Bv[f][ks][0], Bv[f][ks][1])), "v"(A[(m) & 3][ks]) : "memory")
#define MF2(m, f, Bv) asm volatile("v_mfma_f32_16x16x32_bf16 %0, %1, %2, %0\n\tv_mfma_f32_16x16x32_bf16 %0, %3, %4, %0" : "+v"(acc[m][f]) : "v"(cat(Bv[f][0][0], Bv[f][0][1])), "v"(A[(m) & 3][0]), "v"(cat(Bv[f][1][0], Bv[f][1][1])), "v"(A[(m) & 3][1]) : "memory")
    GET_CALL();
    WAITV(0);
    BAR;
    STAMP(1);
    WAITL0; SB0;
#pragma unroll
    for (int i = 0; i < 12; ++i) { RB(B0, i, 0); SB0; }
    RA(0, 0, 0); SB0; RA(0, 1, 0); SB0; RB(B0, 12, 0); SB0; RB(B0, 13, 0); SB0; RA(1, 0, 0); SB0; RA(1, 1, 0); SB0; RB(B0, 14, 0); SB0; RB(B0, 15, 0); SB0; RA(2, 0, 0); SB0; RA(2, 1, 0); SB0;
    BAR;
    int so = 0;
#define CH(m, f, Bc, R) MF(m, f, 0, Bc); R; SB0; MF(m, f, 1, Bc); SB0;
#define GA(m, Bc, Bn, sb, ma, soa) CH(m, 0, Bc, RA(ma, 0, soa)) CH(m, 1, Bc, RA(ma, 1, soa)) CH(m, 2, Bc, RB(Bn, 2 * (m), sb)) CH(m, 3, Bc, RB(Bn, 2 * (m) + 1, sb))
#define GB(m, Bc, Bn, sb, ma, soa) CH(m, 0, Bc, RB(Bn, 2 * (m), sb)) CH(m, 1, Bc, RB(Bn, 2 * (m) + 1, sb)) CH(m, 2, Bc, RA(ma, 0, soa)) CH(m, 3, Bc, RA(ma, 1, soa))
#define KTILE(Bc, Bn, sb) { \
        const int sn = (so == 32768) ? 0 : so + 16384; \
        SB0; \
        GA(0, Bc, Bn, sb, 3, so) GA(1, Bc, Bn, sb, 4, so) GA(2, Bc, Bn, sb, 5, so) GA(3, Bc, Bn, sb, 6, so) GA(4, Bc, Bn, sb, 7, so) \
        CH(5, 0, Bc, RB(Bn, 10, sb)) CH(5, 1, Bc, RB(Bn, 11, sb)) MF2(5, 2, Bc); SB0; \
        asm volatile("" ::: "memory"); __builtin_amdgcn_s_waitcnt(0xC47F);   BAR; SB0; \
        RA(0, 0, sn); SB0; CH(5, 3, Bc, RA(0, 1, sn)) \
        GB(6, Bc, Bn, sb, 1, sn) GB(7, Bc, Bn, sb, 2, sn) \
        so = sn; }
    for (int t = 0; t < NT; t += 4) {
        KTILE(B0, B1, 1)
        KTILE(B1, B0, 2)
        KTILE(B0, B1, 3)
        KTILE(B1, B0, 0)
    }
    asm volatile("s_nop 7\n\ts_nop 7" ::: "memory");
    SB0;
    STAMP(2);
    {
        int mx = 0;
#pragma unroll
        for (int m = 0; m < 8; ++m)
#pragma unroll
            for (int f = 0; f < 4; ++f) mx = imax3(imax3(mx, acc[m][f][0], acc[m][f][1]), acc[m][f][2], acc[m][f][3]);
        const unsigned wmx = wave_umax((unsigned)mx);
        if (lane == 0) xs[wid] = wmx;
        WAITL0;
        BAR;
    }
    const int par = call & 1;
    const int slot = b * 4 + wid;
    char* pws = p.ws[dsel];
    acc[7][3][3] = relu(acc[7][3][3]);
    u32x4 kept;
#pragma unroll
    for (int w = 0; w < 4; ++w) {
        unsigned sg = 0;
#pragma unroll
        for (int i = 31; i >= 0; --i) { const int v = w * 32 + i; sg = __builtin_amdgcn_alignbit(sg, __float_as_uint(acc[v >> 4][(v >> 2) & 3][v & 3]), 31); }
        kept[w] = ~sg;
    }
    {
        auto rm = __builtin_amdgcn_make_buffer_rsrc((void*)(pws + WS_MSK), 0, 2 * MSK_PAR, 0x00020000);
        __builtin_amdgcn_raw_buffer_store_b128(kept, rm, lane * 16, par * MSK_PAR + slot * 1024, 17);
    }
    SB0;
    int base, total;
    pool_scan(__builtin_popcount(kept[0]) + __builtin_popcount(kept[1]) + __builtin_popcount(kept[2]) + __builtin_popcount(kept[3]), fr, fq, base, total);
    const int nch = (int)((wave_umax((unsigned)total) + 63u) >> 6);
    {
        int a = wid * 32768 + fr * 4 + (384 - base) * 64;
#pragma unroll
        for (int v = 0; v < 128; ++v) {
            const float y = acc[v >> 4][(v >> 2) & 3][v & 3];
            *(LAS float*)(lds + a + (127 - v) * 64) = y;
            int t;
            if (v < 127) asm("v_lshrrev_b32 %1, 31, %2\n\tv_lshl_add_u32 %0, %1, 6, %0" : "+v"(a), "=&v"(t) : "v"(y));
        }
        asm volatile("" ::: "memory");
        SB0;
        const int rbase = wid * 32768 + fr * 4 + (48 - 16 * fq) * 64;
        int re = rbase + 64 - (fq & 1) * 64, ro = rbase + (fq & 1) * 64;
        asm volatile("" : "+v"(re), "+v"(ro));
#pragma unroll
        for (int c = 0; c < 8; ++c) {
            if (c < 5 || c < nch) {
#pragma unroll
                for (int i = 0; i < 16; ++i) acc[c][i >> 2][i & 3] = *(LAS float*)(lds + ((i & 1) ? ro : re) + (462 + (i & 1) - 64 * c - i) * 64);
            }
        }
    }
    int coff[8];
#pragma unroll
    for (int c = 0; c < 8; ++c) coff[c] = chunk_off(total, fq, c);
    unsigned slo = (fq & 1) ? 0x0c0c0004u : 0x0c0c0400u, shi = (fq & 1) ? 0x00040c0cu : 0x04000c0cu;
    asm volatile("" : "+v"(slo), "+v"(shi));
#pragma unroll
    for (int m = 0; m < 8; ++m) {
        asm volatile("" : "+v"(coff[m]));
#pragma unroll
        for (int f = 0; f < 4; ++f) asm volatile("" : "+v"(acc[m][f]));
    }
    WAITL0;
    if (lane == 0) xs[4 + wid] = 1u;
    auto rd = __builtin_amdgcn_make_buffer_rsrc((void*)(pws + WS_DAT), 0, DAT_BYTES, 0x00020000);
    STAMP(6);
    unsigned tg[4];
#pragma unroll
    for (int i = 0; i < 4; ++i) { tg[i] = tag_word(call, i); asm volatile("" : "+s"(tg[i])); }
    u64 gv;
    {
        const u64* g = (const u64*)(myws + WS_GLB) + (par * NREP + idx) * 16 + (lane & 7);
        const u64 t0 = __builtin_amdgcn_s_memrealtime();
        for (unsigned it = 1;; ++it) {
            gv = __hip_atomic_load(g, __ATOMIC_RELAXED, __HIP_MEMORY_SCOPE_SYSTEM);
            if (__all((unsigned)(gv >> 32) == call)) break;
            if ((it & 63u) == 0u && __builtin_amdgcn_readfirstlane((int)give_up(tmo, call, t0))) return;
            __builtin_amdgcn_s_sleep(1);
        }
    }
    STAMP(3);
    const float amax = __uint_as_float(wave_umax((unsigned)gv));
    const float inv = amax > 0.f ? 127.0f / amax : 0.f;
#define SEND(c) do { u32x4 ov; \
        _Pragma("unroll") for (int f = 0; f < 4; ++f) ov[f] = (quant4(acc[c][f][0], acc[c][f][1], acc[c][f][2], acc[c][f][3], inv, slo, shi) & 0x7f7f7f7fu) | tg[f]; \
        if ((c) >= 1 && (c) <= 3) WAITV(WIN); \
        __builtin_amdgcn_raw_buffer_store_b128(ov, rd, coff[c], slot * 8192, 17); } while (0)
    SEND(0);
    if (nch > 7) SEND(7);
    if (nch > 6) SEND(6);
    if (nch > 5) SEND(5);
    if (nch > 4) SEND(4);
    if (nch > 3) SEND(3);
    if (nch > 2) SEND(2);
    if (nch > 1) SEND(1);
    STAMP(4);
}

extern "C" void kernel_launch(int rank, int nranks, ncclComm_t comm, hipStream_t stream,
                              void* const* d_in, const int* in_sizes, int n_in,
                              void* d_out, int out_size, const int* in_dtypes, int out_dtype,
                              void* const* d_out_all, void* const* d_ws_all, size_t ws_bytes) {
    if (nranks != NR || n_in < 2 || in_sizes[0] != ML * KD || in_sizes[1] != KD * ND || out_size != NR * ML * NSH || ws_bytes < WS_NEED || (unsigned)rank >= (unsigned)NR) {
        fprintf(stderr, "kernel_launch: unexpected problem shape\n");
        return;
    }
    Params p;
    p.x = (const unsigned short*)d_in[0];
    p.w = (const unsigned short*)d_in[1];
    p.out = (char*)d_out;
    for (int d = 0; d < NR; ++d) p.ws[d] = (char*)d_ws_all[(rank + d) & 7];
    p.rank = rank; p.pad = 0;
    hipLaunchKernelGGL(fused_gemm_a2a, dim3(NWG), dim3(512), 0, stream, p);
    HIP_CHECK(hipGetLastError());
}
```

```cpp
#include <hip/hip_runtime.h>
#include <rccl/rccl.h>
#include <cstdio>

#define HIP_CHECK(expr) do { hipError_t _e = (expr); if (_e != hipSuccess) { \
    fprintf(stderr, "HIP error %s at %s:%d\n", hipGetErrorString(_e), __FILE__, __LINE__); \
    return; } } while (0)

#define LAS __attribute__((address_space(3)))
typedef short v4s __attribute__((ext_vector_type(4)));
typedef short v8s __attribute__((ext_vector_type(8)));
typedef float f32x4 __attribute__((ext_vector_type(4)));
typedef unsigned u32x4 __attribute__((ext_vector_type(4)));
typedef unsigned long long u64;

#define KROT 8
#define TMIN 78
#define TMIN2 0
#define TBEG 0
#define TEND 44
#define TSLK 100
#define KLAG 2
#define IORD 0
#define KMODE 0
#define WNT 0
#define OAUX 16
#define WIN 0

namespace {
constexpr int NR = 8;
constexpr int KD = 4096, ND = 16384, ML = 512, NSH = ND / NR;
constexpr int BM = 256, BN = 128, BK = 64, NT = KD / BK;
constexpr int NWG = 256;
constexpr int LDS_B = 0, LDS_A = 65536, LDS_X = 131072, LDS_LUT = LDS_X + 64, LDS_TOT = 163840;
constexpr int NREP = 32;
constexpr int WS_CNT = 0;
constexpr int WS_TMO = 4096;
constexpr int WS_LOC = 8192;
constexpr int WS_GLB = 16384;
constexpr int WS_STAMP = 65536;
constexpr int WS_MSK = 1 << 20;
constexpr int WS_DAT = 3 << 20;
constexpr int MSK_PAR = 1 << 20, DAT_BYTES = 8 << 20;
constexpr size_t WS_NEED = (size_t)WS_DAT + (size_t)DAT_BYTES;
constexpr u64 DEADLINE = 50000000ull;
constexpr int OOB = 0x40000000;

struct Params {
    const unsigned short* x; const unsigned short* w;
    int rank, pad;
    char* out; char* ws[NR];
};
}

#define SB0 __builtin_amdgcn_sched_barrier(0)
#define WAITV(n) asm volatile("s_waitcnt vmcnt(%0)" :: "n"(n) : "memory")
#define WAITL0 do { asm volatile("" ::: "memory"); __builtin_amdgcn_s_waitcnt(0xC07F); asm volatile("" ::: "memory"); } while (0)
#define BAR do { asm volatile("" ::: "memory"); __builtin_amdgcn_s_barrier(); asm volatile("" ::: "memory"); } while (0)
#define STAMP(i) do { } while (0)

__device__ __forceinline__ v8s cat(v4s a, v4s b) { return __builtin_shufflevector(a, b, 0, 1, 2, 3, 4, 5, 6, 7); }
__device__ __forceinline__ int imax3(int a, float b, float c) { return max(max(a, __float_as_int(b)), __float_as_int(c)); }
__device__ __forceinline__ float relu(float a) { return __int_as_float(max(__float_as_int(a), 0)); }
__device__ __forceinline__ unsigned wave_umax(unsigned v) {
    v = max(v, (unsigned)__builtin_amdgcn_update_dpp(0, (int)v, 0xB1, 0xf, 0xf, true));
    v = max(v, (unsigned)__builtin_amdgcn_update_dpp(0, (int)v, 0x4E, 0xf, 0xf, true));
    v = max(v, (unsigned)__builtin_amdgcn_update_dpp(0, (int)v, 0x141, 0xf, 0xf, true));
    v = max(v, (unsigned)__builtin_amdgcn_update_dpp(0, (int)v, 0x140, 0xf, 0xf, true));
    const unsigned a = __builtin_amdgcn_readlane(v, 0), b = __builtin_amdgcn_readlane(v, 16), c = __builtin_amdgcn_readlane(v, 32), d = __builtin_amdgcn_readlane(v, 48);
    return max(max(a, b), max(c, d));
}
__device__ __forceinline__ bool give_up(unsigned* tmo, unsigned call, u64 t0) {
    if (__hip_atomic_load(tmo, __ATOMIC_RELAXED, __HIP_MEMORY_SCOPE_AGENT) == call || __builtin_amdgcn_s_memrealtime() - t0 > DEADLINE) {
        __hip_atomic_store(tmo, call, __ATOMIC_RELAXED, __HIP_MEMORY_SCOPE_AGENT);
        return true;
    }
    return false;
}
__device__ __forceinline__ unsigned quant4(float a, float b, float c, float d, float inv, unsigned slo, unsigned shi) {
    const unsigned q0 = __float_as_uint(fmaf(a, inv, 12582912.0f)), q1 = __float_as_uint(fmaf(b, inv, 12582912.0f));
    const unsigned q2 = __float_as_uint(fmaf(c, inv, 12582912.0f)), q3 = __float_as_uint(fmaf(d, inv, 12582912.0f));
    return __builtin_amdgcn_perm(q1, q0, slo) | __builtin_amdgcn_perm(q3, q2, shi);
}
__device__ __forceinline__ int chunk_off(int ng, int fq, int c) {
    const bool act = ng > 4 * c + fq;
    const u64 bal = __ballot(act);
    const int rank = __builtin_amdgcn_mbcnt_hi((unsigned)(bal >> 32), __builtin_amdgcn_mbcnt_lo((unsigned)bal, 0));
    return act ? c * 1024 + rank * 16 : OOB;
}
__device__ __forceinline__ void pool_scan(int n, int fr, int fq, int& base, int& ng) {
    const int n0 = __shfl(n, fr), n1 = __shfl(n, fr + 16), n2 = __shfl(n, fr + 32), n3 = __shfl(n, fr + 48);
    base = (fq > 0 ? n0 : 0) + (fq > 1 ? n1 : 0) + (fq > 2 ? n2 : 0);
    ng = (((n0 + n1) + (n2 + n3) + 17) * 3641) >> 16;
}
__device__ __forceinline__ unsigned top_bits(u32x4 v) {
    const unsigned lo = __builtin_amdgcn_udot4(v[1] & 0x80808080u, 0x80402010u, __builtin_amdgcn_udot4(v[0] & 0x80808080u, 0x08040201u, 0u, false), false);
    const unsigned hi = __builtin_amdgcn_udot4(v[3] & 0x80808080u, 0x80402010u, __builtin_amdgcn_udot4(v[2] & 0x80808080u, 0x08040201u, 0u, false), false);
    return (lo >> 7) | (hi << 1);
}
__device__ __forceinline__ unsigned call_tag(unsigned call) { const unsigned t = call % 3u + 1u; return ((t & 1u) << 7) | ((t & 2u) << 14); }

__global__ __launch_bounds__(512, 2) void fused_gemm_a2a(Params p) {
    __shared__ __attribute__((aligned(1024))) char smem[LDS_TOT];
    LAS char* lds = (LAS char*)smem;
    volatile LAS unsigned* xs = (volatile LAS unsigned*)(lds + LDS_X);
    const int tid = threadIdx.x, lane = tid & 63;
    const int wid = __builtin_amdgcn_readfirstlane(tid >> 6);
    const int b = blockIdx.x, dsel = b & 7, idx = b >> 3;
    const int rb = idx & 1, cb = idx >> 1;
    const int m0 = rb * BM;
    const int n0 = ((p.rank + dsel) & 7) * NSH + cb * BN;
    auto rx = __builtin_amdgcn_make_buffer_rsrc((void*)p.x, 0, ML * KD * 2, 0x00020000);
    auto rw = __builtin_amdgcn_make_buffer_rsrc((void*)p.w, 0, KD * ND * 2, 0x00020000);
    const int voA = (m0 + wid * 8 + (lane >> 3)) * (KD * 2) + (((lane & 7) ^ ((wid & 1) * 4 + (lane >> 4))) * 16);
    const int voB = (wid * 4 + (lane >> 4)) * (ND * 2) + n0 * 2 + (((lane & 15) ^ (((wid & 2) << 2) | (lane >> 4))) * 16);
    {
        const int kt = (dsel * KROT) & (NT - 1);
#pragma unroll
        for (int j = 0; j < 2; ++j)
            __builtin_amdgcn_raw_ptr_buffer_load_lds(rw, (LAS void*)(lds + LDS_B + wid * 1024 + j * 8192), 16, voB, kt * (BK * ND * 2) + j * (32 * ND * 2), 0, WNT);
#pragma unroll
        for (int j = 0; j < 4; ++j)
            __builtin_amdgcn_raw_ptr_buffer_load_lds(rx, (LAS void*)(lds + LDS_A + (j >> 1) * 49152 + wid * 1024 + (j & 1) * 8192), 16, voA, kt * (BK * 2) + j * (64 * KD * 2), 0, 0);
    }
    SB0;
    char* myws = p.ws[0];
    unsigned* tmo = (unsigned*)(myws + WS_TMO);
    unsigned call;
#define GET_CALL() do { call = *(const volatile __attribute__((address_space(4))) unsigned*)(myws + WS_CNT) + 1u; asm volatile("" : "+s"(call)); } while (0)
    const int fr = lane & 15, fq = lane >> 4;
    STAMP(0);

    if (__builtin_expect(wid >= 4, 1)) {
        const int lw = wid - 4;
        {
            const int vA = voA - 32 * (KD * 2), vB = voB - 16 * (ND * 2);
            const int ksw = (WNT ? cb : (rb ^ cb)) & 1;
            auto kt_of = [&](int t) { return ((t < 2 ? t : t ^ ksw) + dsel * KROT) & (NT - 1); };
            auto issueW = [&](int t) {
                const int so = t < NT ? kt_of(t) * (BK * ND * 2) : OOB;
#pragma unroll
                for (int j = 0; j < 4; ++j)
                    __builtin_amdgcn_raw_ptr_buffer_load_lds(rw, (LAS void*)(lds + LDS_B + (t & 3) * 16384 + lw * 1024 + j * 4096), 16, vB, so + j * (16 * ND * 2), 0, WNT);
            };
            auto issueX = [&](int t, int so) {
                const int kt = kt_of(t);
#pragma unroll
                for (int j = 0; j < 8; ++j)
                    __builtin_amdgcn_raw_ptr_buffer_load_lds(rx, (LAS void*)(lds + LDS_A + (j >> 2) * 49152 + so + lw * 1024 + (j & 3) * 4096), 16, vA, kt * (BK * 2) + j * (32 * KD * 2), 0, 0);
            };
            issueW(1); issueW(2); issueX(1, 16384); issueW(3); issueX(2, 32768);
            GET_CALL();
            WAITV(28); BAR;
            WAITV(24); BAR;
            unsigned tnext = (unsigned)__builtin_amdgcn_s_memrealtime();
            int so = 0;
            for (int t = 0; t < NT - 3; ++t) {
                WAITV(12);
                if (lw == 3 && t >= TBEG && (t < TEND || TMIN2)) while ((int)((unsigned)__builtin_amdgcn_s_memrealtime() - tnext) < 0) __builtin_amdgcn_s_sleep(0);
                BAR;
                if (lw == 3) { const unsigned now = (unsigned)__builtin_amdgcn_s_memrealtime() - TSLK; tnext = (((int)(now - tnext) > 0 || t < TBEG) ? now : tnext) + (t + 1 < TEND ? TMIN : TMIN2); }
                issueW(t + 4); issueX(t + 3, so);
                so = (so == 32768) ? 0 : so + 16384;
            }
            WAITV(12); BAR;
            {
                auto rm = __builtin_amdgcn_make_buffer_rsrc((void*)(myws + WS_MSK), 0, 2 * MSK_PAR, 0x00020000);
                WAITV(0);
                __builtin_amdgcn_raw_buffer_store_b128((u32x4){0u, 0u, 0u, 0u}, rm, lane * 16, ((call & 1) ^ 1) * MSK_PAR + (b * 4 + lw) * 1024, 17);
            }
            BAR;
            BAR;
            WAITV(0);
        }
        const int par = call & 1;
        if (lw == 2) {
            const int nib = lane & 15, al = lane >> 4;
            unsigned sel = 0;
#pragma unroll
            for (int r = 0; r < 4; ++r) sel |= (((nib >> r) & 1) ? (unsigned)(al + __builtin_popcount(nib & ((1 << r) - 1))) : 0x0cu) << (8 * r);
            *(LAS unsigned*)(lds + LDS_LUT + lane * 4) = sel;
            if (lane < 4) xs[4 + lane] = 0u;
            WAITL0;
        }
        BAR;
        if (lw == 0 && lane == 0) {
            const unsigned mx = max(max(xs[0], xs[1]), max(xs[2], xs[3]));
            __hip_atomic_store((u64*)(myws + WS_LOC) + par * NWG + b, ((u64)call << 32) | mx, __ATOMIC_RELAXED, __HIP_MEMORY_SCOPE_AGENT);
        }
        STAMP(1);
        if (lw == 1 && b == 0) {
            char* base = p.ws[0];
#pragma unroll
            for (int r = 1; r < NR; ++r) base = ((lane & 7) == r) ? p.ws[r] : base;
            u64* dst = (u64*)(base + WS_GLB) + (par * NREP + (lane >> 3)) * 16 + p.rank;
            asm volatile("" : "+v"(dst));
            const u64* tb = (const u64*)(myws + WS_LOC) + par * NWG + lane;
            const u64 t0 = __builtin_amdgcn_s_memrealtime();
            unsigned mx; bool dead = false;
            for (unsigned it = 1;; ++it) {
                bool ok = true; mx = 0;
#pragma unroll
                for (int i = 0; i < 4; ++i) { const u64 v = __hip_atomic_load(tb + i * 64, __ATOMIC_RELAXED, __HIP_MEMORY_SCOPE_AGENT); ok &= (unsigned)(v >> 32) == call; mx = max(mx, (unsigned)v); }
                if (__all(ok)) break;
                if ((it & 63u) == 0u && __builtin_amdgcn_readfirstlane((int)give_up(tmo, call, t0))) { dead = true; break; }
                __builtin_amdgcn_s_sleep(1);
            }
            mx = wave_umax(mx);
            if (!dead) {
#pragma unroll
                for (int i = 0; i < NREP / 8; ++i) __hip_atomic_store(dst + i * 128, ((u64)call << 32) | mx, __ATOMIC_RELAXED, __HIP_MEMORY_SCOPE_SYSTEM);
            }
            if (lane == 0) __hip_atomic_store((unsigned*)(myws + WS_CNT), call, __ATOMIC_RELAXED, __HIP_MEMORY_SCOPE_AGENT);
            STAMP(6);
        }
        const int slot = b * 4 + lw;
        const u64 t0 = __builtin_amdgcn_s_memrealtime();
        bool ok = true;
        u32x4 kept;
        {
            auto rm = __builtin_amdgcn_make_buffer_rsrc((void*)(myws + WS_MSK), 0, 2 * MSK_PAR, 0x00020000);
            for (unsigned it = 1;; ++it) {
                kept = __builtin_amdgcn_raw_buffer_load_b128(rm, lane * 16, par * MSK_PAR + slot * 1024, 17);
                if (__all((int)kept[3] < 0)) break;
                if ((it & 63u) == 0u && __builtin_amdgcn_readfirstlane((int)give_up(tmo, call, t0))) { ok = false; break; }
                __builtin_amdgcn_s_sleep(16);
                asm volatile("" ::: "memory");
            }
        }
        STAMP(2);
        const int sbase = lw * 32768 + fr * 4;
        int wa[32]; unsigned sel[32]; int n, ng;
        pool_scan(__builtin_popcount(kept[0]) + __builtin_popcount(kept[1]) + __builtin_popcount(kept[2]) + __builtin_popcount(kept[3]), fr, fq, n, ng);
#pragma unroll
        for (int w = 0; w < 4; ++w) {
#pragma unroll
            for (int j = 0; j < 8; ++j) {
                const int sh = 4 * (j ^ (fq & 1));
                const int pos = n + __builtin_popcount(__builtin_amdgcn_ubfe(kept[w], 0, sh));
                const int nib = __builtin_amdgcn_ubfe(kept[w], sh, 4);
                wa[w * 8 + j] = sbase + (pos >> 2) * 64;
                sel[w * 8 + j] = *(LAS unsigned*)(lds + LDS_LUT + (((pos & 3) << 4) | nib) * 4);
            }
            n += __builtin_popcount(kept[w]);
        }
        int coff[8];
#pragma unroll
        for (int c = 0; c < 8; ++c) coff[c] = chunk_off(ng, fq, c);
        const int nch = (int)((wave_umax((unsigned)ng) + 3u) >> 2);
        const int xbase = sbase + (4 * ng + (fq >> 1)) * 64 + (fq & 1) * 2;
#pragma unroll
        for (int i = 0; i < 32; ++i) { asm volatile("" : "+v"(wa[i])); asm volatile("" : "+v"(sel[i])); }
        const int src = (p.rank - dsel) & 7;
        int o = (src * ML + m0 + (lw >> 1) * 128 + fr) * NSH + cb * BN + (lw & 1) * 64 + fq * 16;
        asm volatile("" : "+v"(o));
        auto ro = __builtin_amdgcn_make_buffer_rsrc((void*)p.out, 0, NR * ML * NSH, 0x00020000);
        const unsigned tag = call_tag(call);
        while (xs[4 + lw] == 0u) __builtin_amdgcn_s_sleep(1);
        asm volatile("" ::: "memory");
        STAMP(3);
        {
            auto rd = __builtin_amdgcn_make_buffer_rsrc((void*)(myws + WS_DAT), 0, DAT_BYTES, 0x00020000);
            for (unsigned it = 1; ok; ++it) {
                const u32x4 v = __builtin_amdgcn_raw_buffer_load_b128(rd, lane == 0 ? 0 : OOB, slot * 8192, 17);
                if (__builtin_amdgcn_readfirstlane((top_bits(v) & 0x8080u) == tag)) break;
                if ((it & 63u) == 0u && __builtin_amdgcn_readfirstlane((int)give_up(tmo, call, t0))) ok = false;
                __builtin_amdgcn_s_sleep(8);
                asm volatile("" ::: "memory");
            }
            STAMP(7);
            for (unsigned it = 1; ok; ++it) {
                u32x4 v[8];
#pragma unroll
                for (int c = 0; c < 8; ++c) if (c < 4 || c < nch) v[c] = __builtin_amdgcn_raw_buffer_load_b128(rd, coff[c], slot * 8192, 17);
                int left = OOB;
#pragma unroll
                for (int c = 0; c < 8; ++c) if (c < 4 || c < nch) {
                    const unsigned h = top_bits(v[c]);
                    if ((h & 0x8080u) == tag) {
#pragma unroll
                        for (int i = 0; i < 4; ++i) *(LAS unsigned*)(lds + sbase + fq * 256 + (c * 16 + i) * 64) = v[c][i] & 0x7f7f7f7fu;
                        *(LAS unsigned short*)(lds + xbase + c * 128) = (unsigned short)(h & 0x7f7fu);
                        __builtin_amdgcn_raw_buffer_store_b128((u32x4){0u, 0u, 0u, 0u}, rd, coff[c], slot * 8192, 17);
                        coff[c] = OOB;
                    }
                    left = min(left, coff[c]);
                }
                if (__all(left == OOB)) break;
                if ((it & 63u) == 0u && __builtin_amdgcn_readfirstlane((int)give_up(tmo, call, t0))) ok = false;
                __builtin_amdgcn_s_sleep(2);
                asm volatile("" ::: "memory");
            }
        }
        STAMP(4);
#pragma unroll
        for (int m = 0; m < 8; ++m) {
            u32x4 ov;
#pragma unroll
            for (int j = 0; j < 4; ++j) {
                const unsigned lo = *(LAS unsigned*)(lds + wa[m * 4 + j]), hi = *(LAS unsigned*)(lds + wa[m * 4 + j] + 64);
                ov[j] = __builtin_amdgcn_perm(hi, lo, sel[m * 4 + j]);
            }
            if (!ok) ov = (u32x4){0x80808080u, 0x80808080u, 0x80808080u, 0x80808080u};
            __builtin_amdgcn_raw_buffer_store_b128(ov, ro, o, m * 16 * NSH, OAUX);
        }
        STAMP(5);
        return;
    }
    const int wm = wid >> 1, wn = wid & 1;
    const int aA0 = LDS_A + wm * 49152 + fr * 128 + ((fq ^ (fr >> 1)) * 16);
    const int aA1 = aA0 ^ 64;
    int aB[4];
    {
        const int q = fr >> 2, pp = lane & 3;
        const int ch = (wn * 8 + 2 * pp) ^ (((fq & 1) << 3) | q);
        const int a0 = LDS_B + (8 * fq + q) * 256 + ch * 16 + (pp & 1) * 8;
#pragma unroll
        for (int f = 0; f < 4; ++f) aB[f] = a0 ^ (8 * f);
    }
    f32x4 acc[8][4];
#pragma unroll
    for (int m = 0; m < 8; ++m)
#pragma unroll
        for (int f = 0; f < 4; ++f) { acc[m][f] = (f32x4){0.f, 0.f, 0.f, 0.f}; asm volatile("" : "+v"(acc[m][f])); }
    v8s A[4][2];
    v4s B0[4][2][2], B1[4][2][2];
#define RA(m, ks, so) A[(m) & 3][ks] = *(LAS v8s*)(lds + ((ks) ? aA1 : aA0) + (so) + (m) * 2048)
#define RB(Bv, i, sb) Bv[(i) >> 2][((i) >> 1) & 1][(i) & 1] = __builtin_amdgcn_ds_read_tr16_b64_v4i16((LAS v4s*)(lds + aB[(i) >> 2] + (sb) * 16384 + (((i) >> 1) & 1) * 8192 + ((i) & 1) * 1024))
#define MF(m, f, ks, Bv) asm volatile("v_mfma_f32_16x16x32_bf16 %0, %1, %2, %0" : "+v"(acc[m][f]) : "v"(cat(Bv[f][ks][0], Bv[f][ks][1])), "v"(A[(m) & 3][ks]) : "memory")
#define MF2(m, f, Bv) asm volatile("v_mfma_f32_16x16x32_bf16 %0, %1, %2, %0\n\tv_mfma_f32_16x16x32_bf16 %0, %3, %4, %0" : "+v"(acc[m][f]) : "v"(cat(Bv[f][0][0], Bv[f][0][1])), "v"(A[(m) & 3][0]), "v"(cat(Bv[f][1][0], Bv[f][1][1])), "v"(A[(m) & 3][1]) : "memory")
    GET_CALL();
    WAITV(0);
    BAR;
    STAMP(1);
    WAITL0; SB0;
#pragma unroll
    for (int i = 0; i < 12; ++i) { RB(B0, i, 0); SB0; }
    RA(0, 0, 0); SB0; RA(0, 1, 0); SB0; RB(B0, 12, 0); SB0; RB(B0, 13, 0); SB0; RA(1, 0, 0); SB0; RA(1, 1, 0); SB0; RB(B0, 14, 0); SB0; RB(B0, 15, 0); SB0; RA(2, 0, 0); SB0; RA(2, 1, 0); SB0;
    BAR;
    int so = 0;
#define CH(m, f, Bc, R) MF(m, f, 0, Bc); R; SB0; MF(m, f, 1, Bc); SB0;
#define GA(m, Bc, Bn, sb, ma, soa) CH(m, 0, Bc, RA(ma, 0, soa)) CH(m, 1, Bc, RA(ma, 1, soa)) CH(m, 2, Bc, RB(Bn, 2 * (m), sb)) CH(m, 3, Bc, RB(Bn, 2 * (m) + 1, sb))
#define GB(m, Bc, Bn, sb, ma, soa) CH(m, 0, Bc, RB(Bn, 2 * (m), sb)) CH(m, 1, Bc, RB(Bn, 2 * (m) + 1, sb)) CH(m, 2, Bc, RA(ma, 0, soa)) CH(m, 3, Bc, RA(ma, 1, soa))
#define KTILE(Bc, Bn, sb) { \
        const int sn = (so == 32768) ? 0 : so + 16384; \
        SB0; \
        GA(0, Bc, Bn, sb, 3, so) GA(1, Bc, Bn, sb, 4, so) GA(2, Bc, Bn, sb, 5, so) GA(3, Bc, Bn, sb, 6, so) GA(4, Bc, Bn, sb, 7, so) \
        CH(5, 0, Bc, RB(Bn, 10, sb)) CH(5, 1, Bc, RB(Bn, 11, sb)) MF2(5, 2, Bc); SB0; \
        asm volatile("" ::: "memory"); __builtin_amdgcn_s_waitcnt(0xC47F);   BAR; SB0; \
        RA(0, 0, sn); SB0; CH(5, 3, Bc, RA(0, 1, sn)) \
        GB(6, Bc, Bn, sb, 1, sn) GB(7, Bc, Bn, sb, 2, sn) \
        so = sn; }
    for (int t = 0; t < NT; t += 4) {
        KTILE(B0, B1, 1)
        KTILE(B1, B0, 2)
        KTILE(B0, B1, 3)
        KTILE(B1, B0, 0)
    }
    asm volatile("s_nop 7\n\ts_nop 7" ::: "memory");
    SB0;
    STAMP(2);
    {
        int mx = 0;
#pragma unroll
        for (int m = 0; m < 8; ++m)
#pragma unroll
            for (int f = 0; f < 4; ++f) mx = imax3(imax3(mx, acc[m][f][0], acc[m][f][1]), acc[m][f][2], acc[m][f][3]);
        const unsigned wmx = wave_umax((unsigned)mx);
        if (lane == 0) xs[wid] = wmx;
        WAITL0;
        BAR;
    }
    const int par = call & 1;
    const int slot = b * 4 + wid;
    char* pws = p.ws[dsel];
    acc[7][3][3] = relu(acc[7][3][3]);
    u32x4 kept;
#pragma unroll
    for (int w = 0; w < 4; ++w) {
        unsigned sg = 0;
#pragma unroll
        for (int i = 31; i >= 0; --i) { const int v = w * 32 + i; sg = __builtin_amdgcn_alignbit(sg, __float_as_uint(acc[v >> 4][(v >> 2) & 3][v & 3]), 31); }
        kept[w] = ~sg;
    }
    {
        auto rm = __builtin_amdgcn_make_buffer_rsrc((void*)(pws + WS_MSK), 0, 2 * MSK_PAR, 0x00020000);
        __builtin_amdgcn_raw_buffer_store_b128(kept, rm, lane * 16, par * MSK_PAR + slot * 1024, 17);
    }
    SB0;
    int base, ng;
    pool_scan(__builtin_popcount(kept[0]) + __builtin_popcount(kept[1]) + __builtin_popcount(kept[2]) + __builtin_popcount(kept[3]), fr, fq, base, ng);
    const int nch = (int)((wave_umax((unsigned)ng) + 3u) >> 2);
    float ex[8][2];
    {
        int a = wid * 32768 + fr * 4 + (384 - base) * 64;
#pragma unroll
        for (int v = 0; v < 128; ++v) {
            const float y = acc[v >> 4][(v >> 2) & 3][v & 3];
            *(LAS float*)(lds + a + (127 - v) * 64) = y;
            int t;
            if (v < 127) asm("v_lshrrev_b32 %1, 31, %2\n\tv_lshl_add_u32 %0, %1, 6, %0" : "+v"(a), "=&v"(t) : "v"(y));
        }
        asm volatile("" ::: "memory");
        SB0;
        const int rbase = wid * 32768 + fr * 4 + (48 - 16 * fq) * 64;
        int re = rbase + 64 - (fq & 1) * 64, ro = rbase + (fq & 1) * 64;
        asm volatile("" : "+v"(re), "+v"(ro));
#pragma unroll
        for (int c = 0; c < 8; ++c) {
            ex[c][0] = 0.f; ex[c][1] = 0.f;
            if (c < 4 || c < nch) {
#pragma unroll
                for (int i = 0; i < 16; ++i) acc[c][i >> 2][i & 3] = *(LAS float*)(lds + ((i & 1) ? ro : re) + (462 + (i & 1) - 64 * c - i) * 64);
                const int xa = wid * 32768 + fr * 4 + (510 - min(16 * ng + 2 * (4 * c + fq), 510)) * 64;
                ex[c][0] = *(LAS float*)(lds + xa + 64); ex[c][1] = *(LAS float*)(lds + xa);
            }
        }
    }
    int coff[8];
#pragma unroll
    for (int c = 0; c < 8; ++c) coff[c] = chunk_off(ng, fq, c);
    unsigned slo = (fq & 1) ? 0x0c0c0004u : 0x0c0c0400u, shi = (fq & 1) ? 0x00040c0cu : 0x04000c0cu;
    asm volatile("" : "+v"(slo), "+v"(shi));
#pragma unroll
    for (int m = 0; m < 8; ++m) {
        asm volatile("" : "+v"(coff[m]), "+v"(ex[m][0]), "+v"(ex[m][1]));
#pragma unroll
        for (int f = 0; f < 4; ++f) asm volatile("" : "+v"(acc[m][f]));
    }
    WAITL0;
    if (lane == 0) xs[4 + wid] = 1u;
    auto rd = __builtin_amdgcn_make_buffer_rsrc((void*)(pws + WS_DAT), 0, DAT_BYTES, 0x00020000);
    STAMP(6);
    unsigned tag7 = call_tag(call) << 7;
    asm volatile("" : "+s"(tag7));
    u64 gv;
    {
        const u64* g = (const u64*)(myws + WS_GLB) + (par * NREP + idx) * 16 + (lane & 7);
        const u64 t0 = __builtin_amdgcn_s_memrealtime();
        for (unsigned it = 1;; ++it) {
            gv = __hip_atomic_load(g, __ATOMIC_RELAXED, __HIP_MEMORY_SCOPE_SYSTEM);
            if (__all((unsigned)(gv >> 32) == call)) break;
            if ((it & 63u) == 0u && __builtin_amdgcn_readfirstlane((int)give_up(tmo, call, t0))) return;
            __builtin_amdgcn_s_sleep(1);
        }
    }
    STAMP(3);
    const float amax = __uint_as_float(wave_umax((unsigned)gv));
    const float inv = amax > 0.f ? 127.0f / amax : 0.f;
#define SEND(c) do { u32x4 ov; \
        const unsigned e0 = __float_as_uint(fmaf(ex[c][0], inv, 12582912.0f)), e1 = __float_as_uint(fmaf(ex[c][1], inv, 12582912.0f)); \
        const unsigned e7 = ((__builtin_amdgcn_perm(e1, e0, 0x0c0c0400u) & 0x7f7fu) << 7) | tag7; \
        _Pragma("unroll") for (int f = 0; f < 4; ++f) { \
            const unsigned top = __umul24((e7 >> (4 * f)) & 0x780u, 0x204081u);        \
            ov[f] = (quant4(acc[c][f][0], acc[c][f][1], acc[c][f][2], acc[c][f][3], inv, slo, shi) & 0x7f7f7f7fu) | (top & 0x80808080u); } \
        if ((c) >= 1 && (c) <= 3) WAITV(WIN); \
        __builtin_amdgcn_raw_buffer_store_b128(ov, rd, coff[c], slot * 8192, 17); } while (0)
    SEND(0);
    if (nch > 7) SEND(7);
    if (nch > 6) SEND(6);
    if (nch > 5) SEND(5);
    if (nch > 4) SEND(4);
    if (nch > 3) SEND(3);
    if (nch > 2) SEND(2);
    if (nch > 1) SEND(1);
    STAMP(4);
}

extern "C" void kernel_launch(int rank, int nranks, ncclComm_t comm, hipStream_t stream,
                              void* const* d_in, const int* in_sizes, int n_in,
                              void* d_out, int out_size, const int* in_dtypes, int out_dtype,
                              void* const* d_out_all, void* const* d_ws_all, size_t ws_bytes) {
    if (nranks != NR || n_in < 2 || in_sizes[0] != ML * KD || in_sizes[1] != KD * ND || out_size != NR * ML * NSH || ws_bytes < WS_NEED || (unsigned)rank >= (unsigned)NR) {
        fprintf(stderr, "kernel_launch: unexpected problem shape\n");
        return;
    }
    Params p;
    p.x = (const unsigned short*)d_in[0];
    p.w = (const unsigned short*)d_in[1];
    p.out = (char*)d_out;
    for (int d = 0; d < NR; ++d) p.ws[d] = (char*)d_ws_all[(rank + d) & 7];
    p.rank = rank; p.pad = 0;
    hipLaunchKernelGGL(fused_gemm_a2a, dim3(NWG), dim3(512), 0, stream, p);
    HIP_CHECK(hipGetLastError());
}
```

```cpp
#include <hip/hip_runtime.h>
#include <rccl/rccl.h>
#include <cstdio>

#define HIP_CHECK(expr) do { hipError_t _e = (expr); if (_e != hipSuccess) { \
    fprintf(stderr, "HIP error %s at %s:%d\n", hipGetErrorString(_e), __FILE__, __LINE__); \
    return; } } while (0)

#define LAS __attribute__((address_space(3)))
typedef short v4s __attribute__((ext_vector_type(4)));
typedef short v8s __attribute__((ext_vector_type(8)));
typedef float f32x4 __attribute__((ext_vector_type(4)));
typedef unsigned u32x4 __attribute__((ext_vector_type(4)));
typedef unsigned long long u64;

#define TMIN 78
#define TEND 44
#define TSLK 100

namespace {
constexpr int NR = 8;
constexpr int KD = 4096, ND = 16384, ML = 512, NSH = ND / NR;
constexpr int BM = 256, BN = 128, BK = 64, NT = KD / BK;
constexpr int NWG = 256;
constexpr int KROT = 8;
constexpr int LDS_B = 0, LDS_A = 65536, LDS_X = 131072, LDS_LUT = LDS_X + 64, LDS_TOT = 163840;
constexpr int NREP = 32;
constexpr int WS_CNT = 0;
constexpr int WS_TMO = 4096;
constexpr int WS_LOC = 8192;
constexpr int WS_GLB = 16384;
constexpr int WS_STAMP = 65536;
constexpr int WS_MSK = 1 << 20;
constexpr int WS_DAT = 3 << 20;
constexpr int MSK_PAR = 1 << 20, DAT_BYTES = 8 << 20;
constexpr size_t WS_NEED = (size_t)WS_DAT + (size_t)DAT_BYTES;
constexpr u64 DEADLINE = 50000000ull;
constexpr int OOB = 0x40000000;

struct Params {
    const unsigned short* x; const unsigned short* w;
    int rank, pad;
    char* out; char* ws[NR];
};
}

#define SB0 __builtin_amdgcn_sched_barrier(0)
#define WAITV(n) asm volatile("s_waitcnt vmcnt(%0)" :: "n"(n) : "memory")
#define WAITL0 do { asm volatile("" ::: "memory"); __builtin_amdgcn_s_waitcnt(0xC07F); asm volatile("" ::: "memory"); } while (0)
#define BAR do { asm volatile("" ::: "memory"); __builtin_amdgcn_s_barrier(); asm volatile("" ::: "memory"); } while (0)
#define STAMP(i) do { } while (0)

__device__ __forceinline__ v8s cat(v4s a, v4s b) { return __builtin_shufflevector(a, b, 0, 1, 2, 3, 4, 5, 6, 7); }
__device__ __forceinline__ int imax3(int a, float b, float c) { return max(max(a, __float_as_int(b)), __float_as_int(c)); }
__device__ __forceinline__ float relu(float a) { return __int_as_float(max(__float_as_int(a), 0)); }
__device__ __forceinline__ unsigned wave_umax(unsigned v) {
    v = max(v, (unsigned)__builtin_amdgcn_update_dpp(0, (int)v, 0xB1, 0xf, 0xf, true));
    v = max(v, (unsigned)__builtin_amdgcn_update_dpp(0, (int)v, 0x4E, 0xf, 0xf, true));
    v = max(v, (unsigned)__builtin_amdgcn_update_dpp(0, (int)v, 0x141, 0xf, 0xf, true));
    v = max(v, (unsigned)__builtin_amdgcn_update_dpp(0, (int)v, 0x140, 0xf, 0xf, true));
    const unsigned a = __builtin_amdgcn_readlane(v, 0), b = __builtin_amdgcn_readlane(v, 16), c = __builtin_amdgcn_readlane(v, 32), d = __builtin_amdgcn_readlane(v, 48);
    return max(max(a, b), max(c, d));
}
__device__ __forceinline__ bool give_up(unsigned* tmo, unsigned call, u64 t0) {
    if (__hip_atomic_load(tmo, __ATOMIC_RELAXED, __HIP_MEMORY_SCOPE_AGENT) == call || __builtin_amdgcn_s_memrealtime() - t0 > DEADLINE) {
        __hip_atomic_store(tmo, call, __ATOMIC_RELAXED, __HIP_MEMORY_SCOPE_AGENT);
        return true;
    }
    return false;
}
__device__ __forceinline__ unsigned quant4(float a, float b, float c, float d, float inv, unsigned slo, unsigned shi) {
    const unsigned q0 = __float_as_uint(fmaf(a, inv, 12582912.0f)), q1 = __float_as_uint(fmaf(b, inv, 12582912.0f));
    const unsigned q2 = __float_as_uint(fmaf(c, inv, 12582912.0f)), q3 = __float_as_uint(fmaf(d, inv, 12582912.0f));
    return __builtin_amdgcn_perm(q1, q0, slo) | __builtin_amdgcn_perm(q3, q2, shi);
}
__device__ __forceinline__ int chunk_off(int ng, int fq, int c) {
    const bool act = ng > 4 * c + fq;
    const u64 bal = __ballot(act);
    const int rank = __builtin_amdgcn_mbcnt_hi((unsigned)(bal >> 32), __builtin_amdgcn_mbcnt_lo((unsigned)bal, 0));
    return act ? c * 1024 + rank * 16 : OOB;
}
__device__ __forceinline__ void pool_scan(int n, int fr, int fq, int& base, int& ng) {
    const int n0 = __shfl(n, fr), n1 = __shfl(n, fr + 16), n2 = __shfl(n, fr + 32), n3 = __shfl(n, fr + 48);
    base = (fq > 0 ? n0 : 0) + (fq > 1 ? n1 : 0) + (fq > 2 ? n2 : 0);
    ng = (((n0 + n1) + (n2 + n3) + 17) * 3641) >> 16;
}
__device__ __forceinline__ unsigned top_bits(u32x4 v) {
    const unsigned lo = __builtin_amdgcn_udot4(v[1] & 0x80808080u, 0x80402010u, __builtin_amdgcn_udot4(v[0] & 0x80808080u, 0x08040201u, 0u, false), false);
    const unsigned hi = __builtin_amdgcn_udot4(v[3] & 0x80808080u, 0x80402010u, __builtin_amdgcn_udot4(v[2] & 0x80808080u, 0x08040201u, 0u, false), false);
    return (lo >> 7) | (hi << 1);
}
__device__ __forceinline__ unsigned call_tag(unsigned call) { const unsigned t = call % 3u + 1u; return ((t & 1u) << 7) | ((t & 2u) << 14); }

__global__ __launch_bounds__(512, 2) void fused_gemm_a2a(Params p) {
    __shared__ __attribute__((aligned(1024))) char smem[LDS_TOT];
    LAS char* lds = (LAS char*)smem;
    volatile LAS unsigned* xs = (volatile LAS unsigned*)(lds + LDS_X);
    const int tid = threadIdx.x, lane = tid & 63;
    const int wid = __builtin_amdgcn_readfirstlane(tid >> 6);
    const int b = blockIdx.x, dsel = b & 7, idx = b >> 3;
    const int rb = idx & 1, cb = idx >> 1;
    const int m0 = rb * BM;
    const int n0 = ((p.rank + dsel) & 7) * NSH + cb * BN;
    auto rx = __builtin_amdgcn_make_buffer_rsrc((void*)p.x, 0, ML * KD * 2, 0x00020000);
    auto rw = __builtin_amdgcn_make_buffer_rsrc((void*)p.w, 0, KD * ND * 2, 0x00020000);
    const int voA = (m0 + wid * 8 + (lane >> 3)) * (KD * 2) + (((lane & 7) ^ ((wid & 1) * 4 + (lane >> 4))) * 16);
    const int voB = (wid * 4 + (lane >> 4)) * (ND * 2) + n0 * 2 + (((lane & 15) ^ (((wid & 2) << 2) | (lane >> 4))) * 16);
    {
        const int kt = (dsel * KROT) & (NT - 1);
#pragma unroll
        for (int j = 0; j < 2; ++j)
            __builtin_amdgcn_raw_ptr_buffer_load_lds(rw, (LAS void*)(lds + LDS_B + wid * 1024 + j * 8192), 16, voB, kt * (BK * ND * 2) + j * (32 * ND * 2), 0, 0);
#pragma unroll
        for (int j = 0; j < 4; ++j)
            __builtin_amdgcn_raw_ptr_buffer_load_lds(rx, (LAS void*)(lds + LDS_A + (j >> 1) * 49152 + wid * 1024 + (j & 1) * 8192), 16, voA, kt * (BK * 2) + j * (64 * KD * 2), 0, 0);
    }
    SB0;
    char* myws = p.ws[0];
    unsigned* tmo = (unsigned*)(myws + WS_TMO);
    unsigned call;
#define GET_CALL() do { call = *(const volatile __attribute__((address_space(4))) unsigned*)(myws + WS_CNT) + 1u; asm volatile("" : "+s"(call)); } while (0)
    const int fr = lane & 15, fq = lane >> 4;
    STAMP(0);

    if (__builtin_expect(wid >= 4, 1)) {
        const int lw = wid - 4;
        {
            const int vA = voA - 32 * (KD * 2), vB = voB - 16 * (ND * 2);
            const int ksw = (rb ^ cb) & 1;
            auto kt_of = [&](int t) { return ((t < 2 ? t : t ^ ksw) + dsel * KROT) & (NT - 1); };
            auto issueW = [&](int t) {
                const int so = kt_of(t) * (BK * ND * 2);
#pragma unroll
                for (int j = 0; j < 4; ++j)
                    __builtin_amdgcn_raw_ptr_buffer_load_lds(rw, (LAS void*)(lds + LDS_B + (t & 3) * 16384 + lw * 1024 + j * 4096), 16, vB, so + j * (16 * ND * 2), 0, 0);
            };
            auto issueX = [&](int t, int so) {
                const int kt = kt_of(t);
#pragma unroll
                for (int j = 0; j < 8; ++j)
                    __builtin_amdgcn_raw_ptr_buffer_load_lds(rx, (LAS void*)(lds + LDS_A + (j >> 2) * 49152 + so + lw * 1024 + (j & 3) * 4096), 16, vA, kt * (BK * 2) + j * (32 * KD * 2), 0, 0);
            };
            issueW(1); issueW(2); issueX(1, 16384); issueW(3); issueX(2, 32768);
            GET_CALL();
            WAITV(28); BAR;
            WAITV(24); BAR;
            unsigned tnext = (unsigned)__builtin_amdgcn_s_memrealtime();
            int so = 0;
            for (int t = 0; t < NT - 4; ++t) {
                WAITV(12);
                const bool pace = TMIN && lw == 3 && t < TEND;
                if (pace) while ((int)((unsigned)__builtin_amdgcn_s_memrealtime() - tnext) < 0) __builtin_amdgcn_s_sleep(0);
                BAR;
                unsigned now = 0;
                if (pace) now = (unsigned)__builtin_amdgcn_s_memrealtime() - TSLK;
                issueW(t + 4); issueX(t + 3, so);
                so = (so == 32768) ? 0 : so + 16384;
                if (pace) tnext = ((int)(now - tnext) > 0 ? now : tnext) + TMIN;
            }
            WAITV(12); BAR;
            issueX(NT - 1, so);
            WAITV(8); BAR;
            {
                auto rm = __builtin_amdgcn_make_buffer_rsrc((void*)(myws + WS_MSK), 0, 2 * MSK_PAR, 0x00020000);
                WAITV(0);
                __builtin_amdgcn_raw_buffer_store_b128((u32x4){0u, 0u, 0u, 0u}, rm, lane * 16, ((call & 1) ^ 1) * MSK_PAR + (b * 4 + lw) * 1024, 17);
            }
            BAR;
            BAR;
            WAITV(0);
        }
        const int par = call & 1;
        if (lw == 2) {
            const int nib = lane & 15, al = lane >> 4;
            unsigned sel = 0;
#pragma unroll
            for (int r = 0; r < 4; ++r) sel |= (((nib >> r) & 1) ? (unsigned)(al + __builtin_popcount(nib & ((1 << r) - 1))) : 0x0cu) << (8 * r);
            *(LAS unsigned*)(lds + LDS_LUT + lane * 4) = sel;
            if (lane < 4) xs[4 + lane] = 0u;
            WAITL0;
        }
        BAR;
        if (lw == 0 && lane == 0) {
            const unsigned mx = max(max(xs[0], xs[1]), max(xs[2], xs[3]));
            __hip_atomic_store((u64*)(myws + WS_LOC) + par * NWG + b, ((u64)call << 32) | mx, __ATOMIC_RELAXED, __HIP_MEMORY_SCOPE_AGENT);
        }
        STAMP(1);
        if (lw == 1 && b == 0) {
            char* base = p.ws[0];
#pragma unroll
            for (int r = 1; r < NR; ++r) base = ((lane & 7) == r) ? p.ws[r] : base;
            u64* dst = (u64*)(base + WS_GLB) + (par * NREP + (lane >> 3)) * 16 + p.rank;
            asm volatile("" : "+v"(dst));
            const u64* tb = (const u64*)(myws + WS_LOC) + par * NWG + lane;
            const u64 t0 = __builtin_amdgcn_s_memrealtime();
            unsigned mx; bool dead = false;
            for (unsigned it = 1;; ++it) {
                bool ok = true; mx = 0;
#pragma unroll
                for (int i = 0; i < 4; ++i) { const u64 v = __hip_atomic_load(tb + i * 64, __ATOMIC_RELAXED, __HIP_MEMORY_SCOPE_AGENT); ok &= (unsigned)(v >> 32) == call; mx = max(mx, (unsigned)v); }
                if (__all(ok)) break;
                if ((it & 63u) == 0u && __builtin_amdgcn_readfirstlane((int)give_up(tmo, call, t0))) { dead = true; break; }
                __builtin_amdgcn_s_sleep(1);
            }
            mx = wave_umax(mx);
            if (!dead) {
#pragma unroll
                for (int i = 0; i < NREP / 8; ++i) __hip_atomic_store(dst + i * 128, ((u64)call << 32) | mx, __ATOMIC_RELAXED, __HIP_MEMORY_SCOPE_SYSTEM);
            }
            if (lane == 0) __hip_atomic_store((unsigned*)(myws + WS_CNT), call, __ATOMIC_RELAXED, __HIP_MEMORY_SCOPE_AGENT);
            STAMP(6);
        }
        const int slot = b * 4 + lw;
        const u64 t0 = __builtin_amdgcn_s_memrealtime();
        bool ok = true;
        u32x4 kept;
        {
            auto rm = __builtin_amdgcn_make_buffer_rsrc((void*)(myws + WS_MSK), 0, 2 * MSK_PAR, 0x00020000);
            for (unsigned it = 1;; ++it) {
                kept = __builtin_amdgcn_raw_buffer_load_b128(rm, lane * 16, par * MSK_PAR + slot * 1024, 17);
                if (__all((int)kept[3] < 0)) break;
                if ((it & 63u) == 0u && __builtin_amdgcn_readfirstlane((int)give_up(tmo, call, t0))) { ok = false; break; }
                __builtin_amdgcn_s_sleep(16);
                asm volatile("" ::: "memory");
            }
        }
        STAMP(2);
        const int sbase = lw * 32768 + fr * 4;
        int wa[32]; unsigned sel[32]; int n, ng;
        pool_scan(__builtin_popcount(kept[0]) + __builtin_popcount(kept[1]) + __builtin_popcount(kept[2]) + __builtin_popcount(kept[3]), fr, fq, n, ng);
#pragma unroll
        for (int w = 0; w < 4; ++w) {
#pragma unroll
            for (int j = 0; j < 8; ++j) {
                const int sh = 4 * (j ^ (fq & 1));
                const int pos = n + __builtin_popcount(__builtin_amdgcn_ubfe(kept[w], 0, sh));
                const int nib = __builtin_amdgcn_ubfe(kept[w], sh, 4);
                wa[w * 8 + j] = sbase + (pos >> 2) * 64;
                sel[w * 8 + j] = *(LAS unsigned*)(lds + LDS_LUT + (((pos & 3) << 4) | nib) * 4);
            }
            n += __builtin_popcount(kept[w]);
        }
        int coff[8];
#pragma unroll
        for (int c = 0; c < 8; ++c) coff[c] = chunk_off(ng, fq, c);
        const int nch = (int)((wave_umax((unsigned)ng) + 3u) >> 2);
        const int xbase = sbase + (4 * ng + (fq >> 1)) * 64 + (fq & 1) * 2;
#pragma unroll
        for (int i = 0; i < 32; ++i) { asm volatile("" : "+v"(wa[i])); asm volatile("" : "+v"(sel[i])); }
        const int src = (p.rank - dsel) & 7;
        int o = (src * ML + m0 + (lw >> 1) * 128 + fr) * NSH + cb * BN + (lw & 1) * 64 + fq * 16;
        asm volatile("" : "+v"(o));
        auto ro = __builtin_amdgcn_make_buffer_rsrc((void*)p.out, 0, NR * ML * NSH, 0x00020000);
        const unsigned tag = call_tag(call);
        while (xs[4 + lw] == 0u) __builtin_amdgcn_s_sleep(1);
        asm volatile("" ::: "memory");
        STAMP(3);
        {
            auto rd = __builtin_amdgcn_make_buffer_rsrc((void*)(myws + WS_DAT), 0, DAT_BYTES, 0x00020000);
            for (unsigned it = 1; ok; ++it) {
                const u32x4 v = __builtin_amdgcn_raw_buffer_load_b128(rd, lane == 0 ? 0 : OOB, slot * 8192, 17);
                if (__builtin_amdgcn_readfirstlane((top_bits(v) & 0x8080u) == tag)) break;
                if ((it & 63u) == 0u && __builtin_amdgcn_readfirstlane((int)give_up(tmo, call, t0))) ok = false;
                __builtin_amdgcn_s_sleep(8);
                asm volatile("" ::: "memory");
            }
            STAMP(7);
            for (unsigned it = 1; ok; ++it) {
                u32x4 v[8];
#pragma unroll
                for (int c = 0; c < 8; ++c) if (c < 4 || c < nch) v[c] = __builtin_amdgcn_raw_buffer_load_b128(rd, coff[c], slot * 8192, 17);
                int left = OOB;
#pragma unroll
                for (int c = 0; c < 8; ++c) if (c < 4 || c < nch) {
                    const unsigned h = top_bits(v[c]);
                    if ((h & 0x8080u) == tag) {
#pragma unroll
                        for (int i = 0; i < 4; ++i) *(LAS unsigned*)(lds + sbase + fq * 256 + (c * 16 + i) * 64) = v[c][i] & 0x7f7f7f7fu;
                        *(LAS unsigned short*)(lds + xbase + c * 128) = (unsigned short)(h & 0x7f7fu);
                        __builtin_amdgcn_raw_buffer_store_b128((u32x4){0u, 0u, 0u, 0u}, rd, coff[c], slot * 8192, 17);
                        coff[c] = OOB;
                    }
                    left = min(left, coff[c]);
                }
                if (__all(left == OOB)) break;
                if ((it & 63u) == 0u && __builtin_amdgcn_readfirstlane((int)give_up(tmo, call, t0))) ok = false;
                __builtin_amdgcn_s_sleep(0);
                asm volatile("" ::: "memory");
            }
        }
        STAMP(4);
#pragma unroll
        for (int m = 0; m < 8; ++m) {
            u32x4 ov;
#pragma unroll
            for (int j = 0; j < 4; ++j) {
                const unsigned lo = *(LAS unsigned*)(lds + wa[m * 4 + j]), hi = *(LAS unsigned*)(lds + wa[m * 4 + j] + 64);
                ov[j] = __builtin_amdgcn_perm(hi, lo, sel[m * 4 + j]);
            }
            if (!ok) ov = (u32x4){0x80808080u, 0x80808080u, 0x80808080u, 0x80808080u};
            __builtin_amdgcn_raw_buffer_store_b128(ov, ro, o, m * 16 * NSH, 16);
        }
        STAMP(5);
        return;
    }
    const int wm = wid >> 1, wn = wid & 1;
    const int aA0 = LDS_A + wm * 49152 + fr * 128 + ((fq ^ (fr >> 1)) * 16);
    const int aA1 = aA0 ^ 64;
    int aB[4];
    {
        const int q = fr >> 2, pp = lane & 3;
        const int ch = (wn * 8 + 2 * pp) ^ (((fq & 1) << 3) | q);
        const int a0 = LDS_B + (8 * fq + q) * 256 + ch * 16 + (pp & 1) * 8;
#pragma unroll
        for (int f = 0; f < 4; ++f) aB[f] = a0 ^ (8 * f);
    }
    f32x4 acc[8][4];
#pragma unroll
    for (int m = 0; m < 8; ++m)
#pragma unroll
        for (int f = 0; f < 4; ++f) { acc[m][f] = (f32x4){0.f, 0.f, 0.f, 0.f}; asm volatile("" : "+v"(acc[m][f])); }
    v8s A[4][2];
    v4s B0[4][2][2], B1[4][2][2];
#define RA(m, ks, so) A[(m) & 3][ks] = *(LAS v8s*)(lds + ((ks) ? aA1 : aA0) + (so) + (m) * 2048)
#define RB(Bv, i, sb) Bv[(i) >> 2][((i) >> 1) & 1][(i) & 1] = __builtin_amdgcn_ds_read_tr16_b64_v4i16((LAS v4s*)(lds + aB[(i) >> 2] + (sb) * 16384 + (((i) >> 1) & 1) * 8192 + ((i) & 1) * 1024))
#define MF(m, f, ks, Bv) asm volatile("v_mfma_f32_16x16x32_bf16 %0, %1, %2, %0" : "+v"(acc[m][f]) : "v"(cat(Bv[f][ks][0], Bv[f][ks][1])), "v"(A[(m) & 3][ks]) : "memory")
#define MF2(m, f, Bv) asm volatile("v_mfma_f32_16x16x32_bf16 %0, %1, %2, %0\n\tv_mfma_f32_16x16x32_bf16 %0, %3, %4, %0" : "+v"(acc[m][f]) : "v"(cat(Bv[f][0][0], Bv[f][0][1])), "v"(A[(m) & 3][0]), "v"(cat(Bv[f][1][0], Bv[f][1][1])), "v"(A[(m) & 3][1]) : "memory")
    GET_CALL();
    WAITV(0);
    BAR;
    STAMP(1);
    WAITL0; SB0;
#pragma unroll
    for (int i = 0; i < 12; ++i) { RB(B0, i, 0); SB0; }
    RA(0, 0, 0); SB0; RA(0, 1, 0); SB0; RB(B0, 12, 0); SB0; RB(B0, 13, 0); SB0; RA(1, 0, 0); SB0; RA(1, 1, 0); SB0; RB(B0, 14, 0); SB0; RB(B0, 15, 0); SB0; RA(2, 0, 0); SB0; RA(2, 1, 0); SB0;
    BAR;
    int so = 0;
#define CH(m, f, Bc, R) MF(m, f, 0, Bc); R; SB0; MF(m, f, 1, Bc); SB0;
#define GA(m, Bc, Bn, sb, ma, soa) CH(m, 0, Bc, RA(ma, 0, soa)) CH(m, 1, Bc, RA(ma, 1, soa)) CH(m, 2, Bc, RB(Bn, 2 * (m), sb)) CH(m, 3, Bc, RB(Bn, 2 * (m) + 1, sb))
#define GB(m, Bc, Bn, sb, ma, soa) CH(m, 0, Bc, RB(Bn, 2 * (m), sb)) CH(m, 1, Bc, RB(Bn, 2 * (m) + 1, sb)) CH(m, 2, Bc, RA(ma, 0, soa)) CH(m, 3, Bc, RA(ma, 1, soa))
#define KTILE(Bc, Bn, sb) { \
        const int sn = (so == 32768) ? 0 : so + 16384; \
        SB0; \
        GA(0, Bc, Bn, sb, 3, so) GA(1, Bc, Bn, sb, 4, so) GA(2, Bc, Bn, sb, 5, so) GA(3, Bc, Bn, sb, 6, so) GA(4, Bc, Bn, sb, 7, so) \
        CH(5, 0, Bc, RB(Bn, 10, sb)) CH(5, 1, Bc, RB(Bn, 11, sb)) MF2(5, 2, Bc); SB0; \
        asm volatile("" ::: "memory"); __builtin_amdgcn_s_waitcnt(0xC47F);   BAR; SB0; \
        RA(0, 0, sn); SB0; CH(5, 3, Bc, RA(0, 1, sn)) \
        GB(6, Bc, Bn, sb, 1, sn) GB(7, Bc, Bn, sb, 2, sn) \
        so = sn; }
    KTILE(B0, B1, 1)
    KTILE(B1, B0, 2)
    KTILE(B0, B1, 3)
    for (int t = 3; t < NT - 1; t += 4) {
        KTILE(B1, B0, 0)
        KTILE(B0, B1, 1)
        KTILE(B1, B0, 2)
        KTILE(B0, B1, 3)
    }
    int mx = 0;
    unsigned sg[4] = {0u, 0u, 0u, 0u};
#define EPA(m, f) asm volatile("v_max3_i32 %0, %0, %2, %3\n\tv_alignbit_b32 %1, %1, %2, 31\n\tv_alignbit_b32 %1, %1, %3, 31" : "+v"(mx), "+v"(sg[(m) >> 1]) : "v"(acc[m][f][0]), "v"(acc[m][f][1]))
#define EPB(m, f) asm volatile("v_max3_i32 %0, %0, %2, %3\n\tv_alignbit_b32 %1, %1, %2, 31\n\tv_alignbit_b32 %1, %1, %3, 31" : "+v"(mx), "+v"(sg[(m) >> 1]) : "v"(acc[m][f][2]), "v"(acc[m][f][3]))
#define CE(m, f) MF(m, f, 0, B1); EPA((m) - 1, f); SB0; MF(m, f, 1, B1); EPB((m) - 1, f); SB0;
#define GE(m) RA((m) + 3, 0, so); RA((m) + 3, 1, so); SB0; CE(m, 0) CE(m, 1) CE(m, 2) CE(m, 3)
    SB0;
    CH(0, 0, B1, RA(3, 0, so)) CH(0, 1, B1, RA(3, 1, so)) MF2(0, 2, B1); MF2(0, 3, B1); SB0;
    GE(1) GE(2) GE(3) GE(4)
    CE(5, 0) CE(5, 1)
    WAITL0; BAR; SB0;
    CE(5, 2) CE(5, 3) CE(6, 0) CE(6, 1) CE(6, 2) CE(6, 3) CE(7, 0) CE(7, 1) CE(7, 2) CE(7, 3)
    asm volatile("s_nop 7\n\ts_nop 7" ::: "memory");
    SB0;
    STAMP(2);
    {
#pragma unroll
        for (int f = 0; f < 4; ++f) mx = imax3(imax3(mx, acc[7][f][0], acc[7][f][1]), acc[7][f][2], acc[7][f][3]);
        const unsigned wmx = wave_umax((unsigned)mx);
        if (lane == 0) xs[wid] = wmx;
        WAITL0;
        BAR;
    }
    const int par = call & 1;
    const int slot = b * 4 + wid;
    char* pws = p.ws[dsel];
    acc[7][3][3] = relu(acc[7][3][3]);
#pragma unroll
    for (int i = 0; i < 16; ++i) sg[3] = __builtin_amdgcn_alignbit(sg[3], __float_as_uint(acc[7][i >> 2][i & 3]), 31);
    u32x4 kept;
#pragma unroll
    for (int w = 0; w < 4; ++w) kept[w] = ~__builtin_bitreverse32(sg[w]);
    auto rm = __builtin_amdgcn_make_buffer_rsrc((void*)(pws + WS_MSK), 0, 2 * MSK_PAR, 0x00020000);
#define SEND_MAP(k) do { if (wid == (k)) __builtin_amdgcn_raw_buffer_store_b128(kept, rm, lane * 16, par * MSK_PAR + slot * 1024, 17); } while (0)
    int base, ng;
    pool_scan(__builtin_popcount(kept[0]) + __builtin_popcount(kept[1]) + __builtin_popcount(kept[2]) + __builtin_popcount(kept[3]), fr, fq, base, ng);
    const int nch = (int)((wave_umax((unsigned)ng) + 3u) >> 2);
    float ex[8][2];
    {
        int a = wid * 32768 + fr * 4 + (384 - base) * 64;
#pragma unroll
        for (int v = 0; v < 128; ++v) {
            if (v % 40 == 0) SEND_MAP(v / 40);
            const float y = acc[v >> 4][(v >> 2) & 3][v & 3];
            *(LAS float*)(lds + a + (127 - v) * 64) = y;
            int t;
            if (v < 127) asm("v_lshrrev_b32 %1, 31, %2\n\tv_lshl_add_u32 %0, %1, 6, %0" : "+v"(a), "=&v"(t) : "v"(y));
        }
        asm volatile("" ::: "memory");
        SB0;
        const int rbase = wid * 32768 + fr * 4 + (48 - 16 * fq) * 64;
        int re = rbase + 64 - (fq & 1) * 64, ro = rbase + (fq & 1) * 64;
        asm volatile("" : "+v"(re), "+v"(ro));
#pragma unroll
        for (int c = 0; c < 8; ++c) {
            ex[c][0] = 0.f; ex[c][1] = 0.f;
            if (c < 4 || c < nch) {
#pragma unroll
                for (int i = 0; i < 16; ++i) acc[c][i >> 2][i & 3] = *(LAS float*)(lds + ((i & 1) ? ro : re) + (462 + (i & 1) - 64 * c - i) * 64);
                const int xa = wid * 32768 + fr * 4 + (510 - min(16 * ng + 2 * (4 * c + fq), 510)) * 64;
                ex[c][0] = *(LAS float*)(lds + xa + 64); ex[c][1] = *(LAS float*)(lds + xa);
            }
        }
    }
    int coff[8];
#pragma unroll
    for (int c = 0; c < 8; ++c) coff[c] = chunk_off(ng, fq, c);
    unsigned slo = (fq & 1) ? 0x0c0c0004u : 0x0c0c0400u, shi = (fq & 1) ? 0x00040c0cu : 0x04000c0cu;
    asm volatile("" : "+v"(slo), "+v"(shi));
#pragma unroll
    for (int m = 0; m < 8; ++m) {
        asm volatile("" : "+v"(coff[m]), "+v"(ex[m][0]), "+v"(ex[m][1]));
#pragma unroll
        for (int f = 0; f < 4; ++f) asm volatile("" : "+v"(acc[m][f]));
    }
    WAITL0;
    if (lane == 0) xs[4 + wid] = 1u;
    auto rd = __builtin_amdgcn_make_buffer_rsrc((void*)(pws + WS_DAT), 0, DAT_BYTES, 0x00020000);
    STAMP(6);
    unsigned tag7 = call_tag(call) << 7;
    asm volatile("" : "+s"(tag7));
    u64 gv;
    {
        const u64* g = (const u64*)(myws + WS_GLB) + (par * NREP + idx) * 16 + (lane & 7);
        const u64 t0 = __builtin_amdgcn_s_memrealtime();
        for (unsigned it = 1;; ++it) {
            gv = __hip_atomic_load(g, __ATOMIC_RELAXED, __HIP_MEMORY_SCOPE_SYSTEM);
            if (__all((unsigned)(gv >> 32) == call)) break;
            if ((it & 63u) == 0u && __builtin_amdgcn_readfirstlane((int)give_up(tmo, call, t0))) return;
            __builtin_amdgcn_s_sleep(1);
        }
    }
    STAMP(3);
    const float amax = __uint_as_float(wave_umax((unsigned)gv));
    const float inv = amax > 0.f ? 127.0f / amax : 0.f;
#define SEND(c) do { u32x4 ov; \
        const unsigned e0 = __float_as_uint(fmaf(ex[c][0], inv, 12582912.0f)), e1 = __float_as_uint(fmaf(ex[c][1], inv, 12582912.0f)); \
        const unsigned e7 = ((__builtin_amdgcn_perm(e1, e0, 0x0c0c0400u) & 0x7f7fu) << 7) | tag7; \
        _Pragma("unroll") for (int f = 0; f < 4; ++f) { \
            const unsigned top = __umul24((e7 >> (4 * f)) & 0x780u, 0x204081u);        \
            ov[f] = (quant4(acc[c][f][0], acc[c][f][1], acc[c][f][2], acc[c][f][3], inv, slo, shi) & 0x7f7f7f7fu) | (top & 0x80808080u); } \
        if ((c) >= 1 && (c) <= 3) WAITV(0);        \
        __builtin_amdgcn_raw_buffer_store_b128(ov, rd, coff[c], slot * 8192, 17); } while (0)
    SEND(0);
    if (nch > 7) SEND(7);
    if (nch > 6) SEND(6);
    if (nch > 5) SEND(5);
    if (nch > 4) SEND(4);
    if (nch > 3) SEND(3);
    if (nch > 2) SEND(2);
    if (nch > 1) SEND(1);
    STAMP(4);
}

extern "C" void kernel_launch(int rank, int nranks, ncclComm_t comm, hipStream_t stream,
                              void* const* d_in, const int* in_sizes, int n_in,
                              void* d_out, int out_size, const int* in_dtypes, int out_dtype,
                              void* const* d_out_all, void* const* d_ws_all, size_t ws_bytes) {
    if (nranks != NR || n_in < 2 || in_sizes[0] != ML * KD || in_sizes[1] != KD * ND || out_size != NR * ML * NSH || ws_bytes < WS_NEED || (unsigned)rank >= (unsigned)NR) {
        fprintf(stderr, "kernel_launch: unexpected problem shape\n");
        return;
    }
    Params p;
    p.x = (const unsigned short*)d_in[0];
    p.w = (const unsigned short*)d_in[1];
    p.out = (char*)d_out;
    for (int d = 0; d < NR; ++d) p.ws[d] = (char*)d_ws_all[(rank + d) & 7];
    p.rank = rank; p.pad = 0;
    hipLaunchKernelGGL(fused_gemm_a2a, dim3(NWG), dim3(512), 0, stream, p);
    HIP_CHECK(hipGetLastError());
}
```

```cpp
#include <hip/hip_runtime.h>
#include <rccl/rccl.h>
#include <cstdio>

#define HIP_CHECK(expr) do { hipError_t _e = (expr); if (_e != hipSuccess) { \
    fprintf(stderr, "HIP error %s at %s:%d\n", hipGetErrorString(_e), __FILE__, __LINE__); \
    return; } } while (0)

#define LAS __attribute__((address_space(3)))
typedef short v4s __attribute__((ext_vector_type(4)));
typedef short v8s __attribute__((ext_vector_type(8)));
typedef float f32x4 __attribute__((ext_vector_type(4)));
typedef unsigned u32x4 __attribute__((ext_vector_type(4)));
typedef unsigned long long u64;

#define TMIN 78
#define TEND 44
#define TPH 20
#define TSLK 200

namespace {
constexpr int NR = 8;
constexpr int KD = 4096, ND = 16384, ML = 512, NSH = ND / NR;
constexpr int BM = 256, BN = 128, BK = 64, NT = KD / BK;
constexpr int NWG = 256;
constexpr int KROT = 8;
constexpr int LDS_B = 0, LDS_A = 65536, LDS_X = 131072, LDS_LUT = LDS_X + 64, LDS_TOT = 163840;
constexpr int NREP = 32;
constexpr int WS_CNT = 0;
constexpr int WS_TMO = 4096;
constexpr int WS_LOC = 8192;
constexpr int WS_GLB = 16384;
constexpr int WS_STAMP = 65536;
constexpr int WS_MSK = 1 << 20;
constexpr int WS_DAT = 3 << 20;
constexpr int MSK_PAR = 1 << 20, DAT_BYTES = 8 << 20;
constexpr size_t WS_NEED = (size_t)WS_DAT + (size_t)DAT_BYTES;
constexpr u64 DEADLINE = 150000000ull;
constexpr int OOB = 0x40000000;

struct Params {
    const unsigned short* x; const unsigned short* w;
    int rank, pad;
    char* out; char* ws[NR];
};
}

#define SB0 __builtin_amdgcn_sched_barrier(0)
#define WAITV(n) asm volatile("s_waitcnt vmcnt(%0)" :: "n"(n) : "memory")
#define WAITL0 do { asm volatile("" ::: "memory"); __builtin_amdgcn_s_waitcnt(0xC07F); asm volatile("" ::: "memory"); } while (0)
#define BAR do { asm volatile("" ::: "memory"); __builtin_amdgcn_s_barrier(); asm volatile("" ::: "memory"); } while (0)
#define STAMP(i) do { } while (0)

__device__ __forceinline__ v8s cat(v4s a, v4s b) { return __builtin_shufflevector(a, b, 0, 1, 2, 3, 4, 5, 6, 7); }
__device__ __forceinline__ int imax3(int a, float b, float c) { return max(max(a, __float_as_int(b)), __float_as_int(c)); }
__device__ __forceinline__ float relu(float a) { return __int_as_float(max(__float_as_int(a), 0)); }
__device__ __forceinline__ unsigned wave_umax(unsigned v) {
    v = max(v, (unsigned)__builtin_amdgcn_update_dpp(0, (int)v, 0xB1, 0xf, 0xf, true));
    v = max(v, (unsigned)__builtin_amdgcn_update_dpp(0, (int)v, 0x4E, 0xf, 0xf, true));
    v = max(v, (unsigned)__builtin_amdgcn_update_dpp(0, (int)v, 0x141, 0xf, 0xf, true));
    v = max(v, (unsigned)__builtin_amdgcn_update_dpp(0, (int)v, 0x140, 0xf, 0xf, true));
    const unsigned a = __builtin_amdgcn_readlane(v, 0), b = __builtin_amdgcn_readlane(v, 16), c = __builtin_amdgcn_readlane(v, 32), d = __builtin_amdgcn_readlane(v, 48);
    return max(max(a, b), max(c, d));
}
__device__ __forceinline__ bool give_up(unsigned* tmo, unsigned call, u64 t0) {
    if (__hip_atomic_load(tmo, __ATOMIC_RELAXED, __HIP_MEMORY_SCOPE_AGENT) == call || __builtin_amdgcn_s_memrealtime() - t0 > DEADLINE) {
        __hip_atomic_store(tmo, call, __ATOMIC_RELAXED, __HIP_MEMORY_SCOPE_AGENT);
        return true;
    }
    return false;
}
__device__ __forceinline__ unsigned quant4(float a, float b, float c, float d, float inv, unsigned slo, unsigned shi) {
    const unsigned q0 = __float_as_uint(fmaf(a, inv, 12582912.0f)), q1 = __float_as_uint(fmaf(b, inv, 12582912.0f));
    const unsigned q2 = __float_as_uint(fmaf(c, inv, 12582912.0f)), q3 = __float_as_uint(fmaf(d, inv, 12582912.0f));
    return __builtin_amdgcn_perm(q1, q0, slo) | __builtin_amdgcn_perm(q3, q2, shi);
}
__device__ __forceinline__ int chunk_off(int ng, int fq, int c) {
    const bool act = ng > 4 * c + fq;
    const u64 bal = __ballot(act);
    const int rank = __builtin_amdgcn_mbcnt_hi((unsigned)(bal >> 32), __builtin_amdgcn_mbcnt_lo((unsigned)bal, 0));
    return act ? c * 1024 + rank * 16 : OOB;
}
__device__ __forceinline__ void pool_scan(int n, int fr, int fq, int& base, int& ng) {
    const int n0 = __shfl(n, fr), n1 = __shfl(n, fr + 16), n2 = __shfl(n, fr + 32), n3 = __shfl(n, fr + 48);
    base = (fq > 0 ? n0 : 0) + (fq > 1 ? n1 : 0) + (fq > 2 ? n2 : 0);
    ng = (((n0 + n1) + (n2 + n3) + 17) * 3641) >> 16;
}
__device__ __forceinline__ unsigned top_bits(u32x4 v) {
    const unsigned lo = __builtin_amdgcn_udot4(v[1] & 0x80808080u, 0x80402010u, __builtin_amdgcn_udot4(v[0] & 0x80808080u, 0x08040201u, 0u, false), false);
    const unsigned hi = __builtin_amdgcn_udot4(v[3] & 0x80808080u, 0x80402010u, __builtin_amdgcn_udot4(v[2] & 0x80808080u, 0x08040201u, 0u, false), false);
    return (lo >> 7) | (hi << 1);
}
__device__ __forceinline__ unsigned call_tag(unsigned call) { const unsigned t = call % 3u + 1u; return ((t & 1u) << 7) | ((t & 2u) << 14); }

__global__ __launch_bounds__(512, 2) void fused_gemm_a2a(Params p) {
    __shared__ __attribute__((aligned(1024))) char smem[LDS_TOT];
    LAS char* lds = (LAS char*)smem;
    volatile LAS unsigned* xs = (volatile LAS unsigned*)(lds + LDS_X);
    const int tid = threadIdx.x, lane = tid & 63;
    const int wid = __builtin_amdgcn_readfirstlane(tid >> 6);
    const int b = blockIdx.x, dsel = b & 7, idx = b >> 3;
    const int rb = idx & 1, cb = idx >> 1;
    const int m0 = rb * BM;
    const int n0 = ((p.rank + dsel) & 7) * NSH + cb * BN;
    auto rx = __builtin_amdgcn_make_buffer_rsrc((void*)p.x, 0, ML * KD * 2, 0x00020000);
    auto rw = __builtin_amdgcn_make_buffer_rsrc((void*)p.w, 0, KD * ND * 2, 0x00020000);
    const int voA = (m0 + wid * 8 + (lane >> 3)) * (KD * 2) + (((lane & 7) ^ ((wid & 1) * 4 + (lane >> 4))) * 16);
    const int voB = (wid * 4 + (lane >> 4)) * (ND * 2) + n0 * 2 + (((lane & 15) ^ (((wid & 2) << 2) | (lane >> 4))) * 16);
    {
        const int kt = (dsel * KROT) & (NT - 1);
#pragma unroll
        for (int j = 0; j < 2; ++j)
            __builtin_amdgcn_raw_ptr_buffer_load_lds(rw, (LAS void*)(lds + LDS_B + wid * 1024 + j * 8192), 16, voB, kt * (BK * ND * 2) + j * (32 * ND * 2), 0, 0);
#pragma unroll
        for (int j = 0; j < 4; ++j)
            __builtin_amdgcn_raw_ptr_buffer_load_lds(rx, (LAS void*)(lds + LDS_A + (j >> 1) * 49152 + wid * 1024 + (j & 1) * 8192), 16, voA, kt * (BK * 2) + j * (64 * KD * 2), 0, 0);
    }
    SB0;
    char* myws = p.ws[0];
    unsigned* tmo = (unsigned*)(myws + WS_TMO);
    unsigned call;
#define GET_CALL() do { call = *(const volatile __attribute__((address_space(4))) unsigned*)(myws + WS_CNT) + 1u; asm volatile("" : "+s"(call)); } while (0)
    const int fr = lane & 15, fq = lane >> 4;
    STAMP(0);

    if (__builtin_expect(wid >= 4, 1)) {
        const int lw = wid - 4;
        {
            const int vA = voA - 32 * (KD * 2), vB = voB - 16 * (ND * 2);
            const int ksw = (rb ^ cb) & 1;
            auto kt_of = [&](int t) { return ((t < 2 ? t : t ^ ksw) + dsel * KROT) & (NT - 1); };
            auto issueW = [&](int t) {
                const int so = kt_of(t) * (BK * ND * 2);
#pragma unroll
                for (int j = 0; j < 4; ++j)
                    __builtin_amdgcn_raw_ptr_buffer_load_lds(rw, (LAS void*)(lds + LDS_B + (t & 3) * 16384 + lw * 1024 + j * 4096), 16, vB, so + j * (16 * ND * 2), 0, 0);
            };
            auto issueX = [&](int t, int so) {
                const int kt = kt_of(t);
#pragma unroll
                for (int j = 0; j < 8; ++j)
                    __builtin_amdgcn_raw_ptr_buffer_load_lds(rx, (LAS void*)(lds + LDS_A + (j >> 2) * 49152 + so + lw * 1024 + (j & 3) * 4096), 16, vA, kt * (BK * 2) + j * (32 * KD * 2), 0, 0);
            };
            issueW(1); issueW(2); issueX(1, 16384); issueW(3); issueX(2, 32768);
            GET_CALL();
            WAITV(28); BAR;
            WAITV(24); BAR;
            unsigned tnext = (unsigned)__builtin_amdgcn_s_memrealtime() - dsel * TPH;
            int so = 0;
            for (int t = 0; t < NT - 4; ++t) {
                WAITV(12);
                const bool pace = TMIN && lw == 3 && t < TEND;
                if (pace) while ((int)((unsigned)__builtin_amdgcn_s_memrealtime() - tnext) < 0) __builtin_amdgcn_s_sleep(0);
                BAR;
                unsigned now = 0;
                if (pace) now = (unsigned)__builtin_amdgcn_s_memrealtime() - TSLK;
                issueW(t + 4); issueX(t + 3, so);
                so = (so == 32768) ? 0 : so + 16384;
                if (pace) tnext = ((int)(now - tnext) > 0 ? now : tnext) + TMIN;
            }
            WAITV(12); BAR;
            issueX(NT - 1, so);
            WAITV(8); BAR;
            {
                auto rm = __builtin_amdgcn_make_buffer_rsrc((void*)(myws + WS_MSK), 0, 2 * MSK_PAR, 0x00020000);
                WAITV(0);
                __builtin_amdgcn_raw_buffer_store_b128((u32x4){0u, 0u, 0u, 0u}, rm, lane * 16, ((call & 1) ^ 1) * MSK_PAR + (b * 4 + lw) * 1024, 17);
            }
            BAR;
            BAR;
            WAITV(0);
        }
        const int par = call & 1;
        if (lw == 2) {
            const int nib = lane & 15, al = lane >> 4;
            unsigned sel = 0;
#pragma unroll
            for (int r = 0; r < 4; ++r) sel |= (((nib >> r) & 1) ? (unsigned)(al + __builtin_popcount(nib & ((1 << r) - 1))) : 0x0cu) << (8 * r);
            *(LAS unsigned*)(lds + LDS_LUT + lane * 4) = sel;
            if (lane < 4) xs[4 + lane] = 0u;
            WAITL0;
        }
        BAR;
        if (lw == 0 && lane == 0) {
            const unsigned mx = max(max(xs[0], xs[1]), max(xs[2], xs[3]));
            __hip_atomic_store((u64*)(myws + WS_LOC) + par * NWG + b, ((u64)call << 32) | mx, __ATOMIC_RELAXED, __HIP_MEMORY_SCOPE_AGENT);
        }
        STAMP(1);
        if (lw == 1 && b == 0) {
            char* base = p.ws[0];
#pragma unroll
            for (int r = 1; r < NR; ++r) base = ((lane & 7) == r) ? p.ws[r] : base;
            u64* dst = (u64*)(base + WS_GLB) + (par * NREP + (lane >> 3)) * 16 + p.rank;
            asm volatile("" : "+v"(dst));
            const u64* tb = (const u64*)(myws + WS_LOC) + par * NWG + lane;
            const u64 t0 = __builtin_amdgcn_s_memrealtime();
            unsigned mx; bool dead = false;
            for (unsigned it = 1;; ++it) {
                bool ok = true; mx = 0;
#pragma unroll
                for (int i = 0; i < 4; ++i) { const u64 v = __hip_atomic_load(tb + i * 64, __ATOMIC_RELAXED, __HIP_MEMORY_SCOPE_AGENT); ok &= (unsigned)(v >> 32) == call; mx = max(mx, (unsigned)v); }
                if (__all(ok)) break;
                if ((it & 63u) == 0u && __builtin_amdgcn_readfirstlane((int)give_up(tmo, call, t0))) { dead = true; break; }
                __builtin_amdgcn_s_sleep(1);
            }
            mx = wave_umax(mx);
            if (!dead) {
#pragma unroll
                for (int i = 0; i < NREP / 8; ++i) __hip_atomic_store(dst + i * 128, ((u64)call << 32) | mx, __ATOMIC_RELAXED, __HIP_MEMORY_SCOPE_SYSTEM);
            }
            if (lane == 0) __hip_atomic_store((unsigned*)(myws + WS_CNT), call, __ATOMIC_RELAXED, __HIP_MEMORY_SCOPE_AGENT);
            STAMP(6);
        }
        const int slot = b * 4 + lw;
        const u64 t0 = __builtin_amdgcn_s_memrealtime();
        bool ok = true;
        u32x4 kept;
        {
            auto rm = __builtin_amdgcn_make_buffer_rsrc((void*)(myws + WS_MSK), 0, 2 * MSK_PAR, 0x00020000);
            for (unsigned it = 1;; ++it) {
                kept = __builtin_amdgcn_raw_buffer_load_b128(rm, lane * 16, par * MSK_PAR + slot * 1024, 17);
                if (__all((int)kept[3] < 0)) break;
                if ((it & 63u) == 0u && __builtin_amdgcn_readfirstlane((int)give_up(tmo, call, t0))) { ok = false; break; }
                __builtin_amdgcn_s_sleep(16);
                asm volatile("" ::: "memory");
            }
        }
        STAMP(2);
        const int sbase = lw * 32768 + fr * 4;
        int wa[32]; unsigned sel[32]; int n, ng;
        pool_scan(__builtin_popcount(kept[0]) + __builtin_popcount(kept[1]) + __builtin_popcount(kept[2]) + __builtin_popcount(kept[3]), fr, fq, n, ng);
#pragma unroll
        for (int w = 0; w < 4; ++w) {
#pragma unroll
            for (int j = 0; j < 8; ++j) {
                const int sh = 4 * (j ^ (fq & 1));
                const int pos = n + __builtin_popcount(__builtin_amdgcn_ubfe(kept[w], 0, sh));
                const int nib = __builtin_amdgcn_ubfe(kept[w], sh, 4);
                wa[w * 8 + j] = sbase + (pos >> 2) * 64;
                sel[w * 8 + j] = *(LAS unsigned*)(lds + LDS_LUT + (((pos & 3) << 4) | nib) * 4);
            }
            n += __builtin_popcount(kept[w]);
        }
        int coff[8];
#pragma unroll
        for (int c = 0; c < 8; ++c) coff[c] = chunk_off(ng, fq, c);
        const int nch = (int)((wave_umax((unsigned)ng) + 3u) >> 2);
        const int xbase = sbase + (4 * ng + (fq >> 1)) * 64 + (fq & 1) * 2;
#pragma unroll
        for (int i = 0; i < 32; ++i) { asm volatile("" : "+v"(wa[i])); asm volatile("" : "+v"(sel[i])); }
        const int src = (p.rank - dsel) & 7;
        int o = (src * ML + m0 + (lw >> 1) * 128 + fr) * NSH + cb * BN + (lw & 1) * 64 + fq * 16;
        asm volatile("" : "+v"(o));
        auto ro = __builtin_amdgcn_make_buffer_rsrc((void*)p.out, 0, NR * ML * NSH, 0x00020000);
        const unsigned tag = call_tag(call);
        while (xs[4 + lw] == 0u) __builtin_amdgcn_s_sleep(1);
        asm volatile("" ::: "memory");
        STAMP(3);
        {
            auto rd = __builtin_amdgcn_make_buffer_rsrc((void*)(myws + WS_DAT), 0, DAT_BYTES, 0x00020000);
            for (unsigned it = 1; ok; ++it) {
                const u32x4 v = __builtin_amdgcn_raw_buffer_load_b128(rd, lane == 0 ? 0 : OOB, slot * 8192, 17);
                if (__builtin_amdgcn_readfirstlane((top_bits(v) & 0x8080u) == tag)) break;
                if ((it & 63u) == 0u && __builtin_amdgcn_readfirstlane((int)give_up(tmo, call, t0))) ok = false;
                __builtin_amdgcn_s_sleep(8);
                asm volatile("" ::: "memory");
            }
            STAMP(7);
            for (unsigned it = 1; ok; ++it) {
                u32x4 v[8];
#pragma unroll
                for (int c = 0; c < 8; ++c) if (c < 4 || c < nch) v[c] = __builtin_amdgcn_raw_buffer_load_b128(rd, coff[c], slot * 8192, 17);
                int left = OOB;
#pragma unroll
                for (int c = 0; c < 8; ++c) if (c < 4 || c < nch) {
                    const unsigned h = top_bits(v[c]);
                    if ((h & 0x8080u) == tag) {
#pragma unroll
                        for (int i = 0; i < 4; ++i) *(LAS unsigned*)(lds + sbase + fq * 256 + (c * 16 + i) * 64) = v[c][i] & 0x7f7f7f7fu;
                        *(LAS unsigned short*)(lds + xbase + c * 128) = (unsigned short)(h & 0x7f7fu);
                        __builtin_amdgcn_raw_buffer_store_b128((u32x4){0u, 0u, 0u, 0u}, rd, coff[c], slot * 8192, 17);
                        coff[c] = OOB;
                    }
                    left = min(left, coff[c]);
                }
                if (__all(left == OOB)) break;
                if ((it & 63u) == 0u && __builtin_amdgcn_readfirstlane((int)give_up(tmo, call, t0))) ok = false;
                __builtin_amdgcn_s_sleep(0);
                asm volatile("" ::: "memory");
            }
        }
        STAMP(4);
#pragma unroll
        for (int m = 0; m < 8; ++m) {
            u32x4 ov;
#pragma unroll
            for (int j = 0; j < 4; ++j) {
                const unsigned lo = *(LAS unsigned*)(lds + wa[m * 4 + j]), hi = *(LAS unsigned*)(lds + wa[m * 4 + j] + 64);
                ov[j] = __builtin_amdgcn_perm(hi, lo, sel[m * 4 + j]);
            }
            if (!ok) ov = (u32x4){0x80808080u, 0x80808080u, 0x80808080u, 0x80808080u};
            __builtin_amdgcn_raw_buffer_store_b128(ov, ro, o, m * 16 * NSH, 16);
        }
        STAMP(5);
        return;
    }
    const int wm = wid >> 1, wn = wid & 1;
    const int aA0 = LDS_A + wm * 49152 + fr * 128 + ((fq ^ (fr >> 1)) * 16);
    const int aA1 = aA0 ^ 64;
    int aB[4];
    {
        const int q = fr >> 2, pp = lane & 3;
        const int ch = (wn * 8 + 2 * pp) ^ (((fq & 1) << 3) | q);
        const int a0 = LDS_B + (8 * fq + q) * 256 + ch * 16 + (pp & 1) * 8;
#pragma unroll
        for (int f = 0; f < 4; ++f) aB[f] = a0 ^ (8 * f);
    }
    f32x4 acc[8][4];
#pragma unroll
    for (int m = 0; m < 8; ++m)
#pragma unroll
        for (int f = 0; f < 4; ++f) { acc[m][f] = (f32x4){0.f, 0.f, 0.f, 0.f}; asm volatile("" : "+v"(acc[m][f])); }
    v8s A[4][2];
    v4s B0[4][2][2], B1[4][2][2];
#define RA(m, ks, so) A[(m) & 3][ks] = *(LAS v8s*)(lds + ((ks) ? aA1 : aA0) + (so) + (m) * 2048)
#define RB(Bv, i, sb) Bv[(i) >> 2][((i) >> 1) & 1][(i) & 1] = __builtin_amdgcn_ds_read_tr16_b64_v4i16((LAS v4s*)(lds + aB[(i) >> 2] + (sb) * 16384 + (((i) >> 1) & 1) * 8192 + ((i) & 1) * 1024))
#define MF(m, f, ks, Bv) asm volatile("v_mfma_f32_16x16x32_bf16 %0, %1, %2, %0" : "+v"(acc[m][f]) : "v"(cat(Bv[f][ks][0], Bv[f][ks][1])), "v"(A[(m) & 3][ks]) : "memory")
#define MF2(m, f, Bv) asm volatile("v_mfma_f32_16x16x32_bf16 %0, %1, %2, %0\n\tv_mfma_f32_16x16x32_bf16 %0, %3, %4, %0" : "+v"(acc[m][f]) : "v"(cat(Bv[f][0][0], Bv[f][0][1])), "v"(A[(m) & 3][0]), "v"(cat(Bv[f][1][0], Bv[f][1][1])), "v"(A[(m) & 3][1]) : "memory")
    GET_CALL();
    WAITV(0);
    BAR;
    STAMP(1);
    WAITL0; SB0;
#pragma unroll
    for (int i = 0; i < 12; ++i) { RB(B0, i, 0); SB0; }
    RA(0, 0, 0); SB0; RA(0, 1, 0); SB0; RB(B0, 12, 0); SB0; RB(B0, 13, 0); SB0; RA(1, 0, 0); SB0; RA(1, 1, 0); SB0; RB(B0, 14, 0); SB0; RB(B0, 15, 0); SB0; RA(2, 0, 0); SB0; RA(2, 1, 0); SB0;
    BAR;
    int so = 0;
#define CH(m, f, Bc, R) MF(m, f, 0, Bc); R; SB0; MF(m, f, 1, Bc); SB0;
#define GA(m, Bc, Bn, sb, ma, soa) CH(m, 0, Bc, RA(ma, 0, soa)) CH(m, 1, Bc, RA(ma, 1, soa)) CH(m, 2, Bc, RB(Bn, 2 * (m), sb)) CH(m, 3, Bc, RB(Bn, 2 * (m) + 1, sb))
#define GB(m, Bc, Bn, sb, ma, soa) CH(m, 0, Bc, RB(Bn, 2 * (m), sb)) CH(m, 1, Bc, RB(Bn, 2 * (m) + 1, sb)) CH(m, 2, Bc, RA(ma, 0, soa)) CH(m, 3, Bc, RA(ma, 1, soa))
#define KTILE(Bc, Bn, sb) { \
        const int sn = (so == 32768) ? 0 : so + 16384; \
        SB0; \
        GA(0, Bc, Bn, sb, 3, so) GA(1, Bc, Bn, sb, 4, so) GA(2, Bc, Bn, sb, 5, so) GA(3, Bc, Bn, sb, 6, so) GA(4, Bc, Bn, sb, 7, so) \
        CH(5, 0, Bc, RB(Bn, 10, sb)) CH(5, 1, Bc, RB(Bn, 11, sb)) MF2(5, 2, Bc); SB0; \
        asm volatile("" ::: "memory"); __builtin_amdgcn_s_waitcnt(0xC47F);   BAR; SB0; \
        RA(0, 0, sn); SB0; CH(5, 3, Bc, RA(0, 1, sn)) \
        GB(6, Bc, Bn, sb, 1, sn) GB(7, Bc, Bn, sb, 2, sn) \
        so = sn; }
    KTILE(B0, B1, 1)
    KTILE(B1, B0, 2)
    KTILE(B0, B1, 3)
    for (int t = 3; t < NT - 1; t += 4) {
        KTILE(B1, B0, 0)
        KTILE(B0, B1, 1)
        KTILE(B1, B0, 2)
        KTILE(B0, B1, 3)
    }
    int mx = 0;
    unsigned sg[4] = {0u, 0u, 0u, 0u};
#define EPA(m, f) asm volatile("v_max3_i32 %0, %0, %2, %3\n\tv_alignbit_b32 %1, %1, %2, 31\n\tv_alignbit_b32 %1, %1, %3, 31" : "+v"(mx), "+v"(sg[(m) >> 1]) : "v"(acc[m][f][0]), "v"(acc[m][f][1]))
#define EPB(m, f) asm volatile("v_max3_i32 %0, %0, %2, %3\n\tv_alignbit_b32 %1, %1, %2, 31\n\tv_alignbit_b32 %1, %1, %3, 31" : "+v"(mx), "+v"(sg[(m) >> 1]) : "v"(acc[m][f][2]), "v"(acc[m][f][3]))
#define CE(m, f) MF(m, f, 0, B1); EPA((m) - 1, f); SB0; MF(m, f, 1, B1); EPB((m) - 1, f); SB0;
#define GE(m) RA((m) + 3, 0, so); RA((m) + 3, 1, so); SB0; CE(m, 0) CE(m, 1) CE(m, 2) CE(m, 3)
    SB0;
    CH(0, 0, B1, RA(3, 0, so)) CH(0, 1, B1, RA(3, 1, so)) MF2(0, 2, B1); MF2(0, 3, B1); SB0;
    GE(1) GE(2) GE(3) GE(4)
    CE(5, 0) CE(5, 1)
    WAITL0; BAR; SB0;
    CE(5, 2) CE(5, 3) CE(6, 0) CE(6, 1) CE(6, 2) CE(6, 3) CE(7, 0) CE(7, 1) CE(7, 2) CE(7, 3)
    asm volatile("s_nop 7\n\ts_nop 7" ::: "memory");
    SB0;
    STAMP(2);
    {
#pragma unroll
        for (int f = 0; f < 4; ++f) mx = imax3(imax3(mx, acc[7][f][0], acc[7][f][1]), acc[7][f][2], acc[7][f][3]);
        const unsigned wmx = wave_umax((unsigned)mx);
        if (lane == 0) xs[wid] = wmx;
        WAITL0;
        BAR;
    }
    const int par = call & 1;
    const int slot = b * 4 + wid;
    char* pws = p.ws[dsel];
    acc[7][3][3] = relu(acc[7][3][3]);
#pragma unroll
    for (int i = 0; i < 16; ++i) sg[3] = __builtin_amdgcn_alignbit(sg[3], __float_as_uint(acc[7][i >> 2][i & 3]), 31);
    u32x4 kept;
#pragma unroll
    for (int w = 0; w < 4; ++w) kept[w] = ~__builtin_bitreverse32(sg[w]);
    auto rm = __builtin_amdgcn_make_buffer_rsrc((void*)(pws + WS_MSK), 0, 2 * MSK_PAR, 0x00020000);
#define SEND_MAP(k) do { if (wid == (k)) __builtin_amdgcn_raw_buffer_store_b128(kept, rm, lane * 16, par * MSK_PAR + slot * 1024, 17); } while (0)
    int base, ng;
    pool_scan(__builtin_popcount(kept[0]) + __builtin_popcount(kept[1]) + __builtin_popcount(kept[2]) + __builtin_popcount(kept[3]), fr, fq, base, ng);
    const int nch = (int)((wave_umax((unsigned)ng) + 3u) >> 2);
    float ex[8][2];
    {
        int a = wid * 32768 + fr * 4 + (384 - base) * 64;
#pragma unroll
        for (int v = 0; v < 128; ++v) {
            if (v % 40 == 0) SEND_MAP(v / 40);
            const float y = acc[v >> 4][(v >> 2) & 3][v & 3];
            *(LAS float*)(lds + a + (127 - v) * 64) = y;
            int t;
            if (v < 127) asm("v_lshrrev_b32 %1, 31, %2\n\tv_lshl_add_u32 %0, %1, 6, %0" : "+v"(a), "=&v"(t) : "v"(y));
        }
        asm volatile("" ::: "memory");
        SB0;
        const int rbase = wid * 32768 + fr * 4 + (48 - 16 * fq) * 64;
        int re = rbase + 64 - (fq & 1) * 64, ro = rbase + (fq & 1) * 64;
        asm volatile("" : "+v"(re), "+v"(ro));
#pragma unroll
        for (int c = 0; c < 8; ++c) {
            ex[c][0] = 0.f; ex[c][1] = 0.f;
            if (c < 4 || c < nch) {
#pragma unroll
                for (int i = 0; i < 16; ++i) acc[c][i >> 2][i & 3] = *(LAS float*)(lds + ((i & 1) ? ro : re) + (462 + (i & 1) - 64 * c - i) * 64);
                const int xa = wid * 32768 + fr * 4 + (510 - min(16 * ng + 2 * (4 * c + fq), 510)) * 64;
                ex[c][0] = *(LAS float*)(lds + xa + 64); ex[c][1] = *(LAS float*)(lds + xa);
            }
        }
    }
    int coff[8];
#pragma unroll
    for (int c = 0; c < 8; ++c) coff[c] = chunk_off(ng, fq, c);
    unsigned slo = (fq & 1) ? 0x0c0c0004u : 0x0c0c0400u, shi = (fq & 1) ? 0x00040c0cu : 0x04000c0cu;
    asm volatile("" : "+v"(slo), "+v"(shi));
#pragma unroll
    for (int m = 0; m < 8; ++m) {
        asm volatile("" : "+v"(coff[m]), "+v"(ex[m][0]), "+v"(ex[m][1]));
#pragma unroll
        for (int f = 0; f < 4; ++f) asm volatile("" : "+v"(acc[m][f]));
    }
    WAITL0;
    if (lane == 0) xs[4 + wid] = 1u;
    auto rd = __builtin_amdgcn_make_buffer_rsrc((void*)(pws + WS_DAT), 0, DAT_BYTES, 0x00020000);
    STAMP(6);
    unsigned tag7 = call_tag(call) << 7;
    asm volatile("" : "+s"(tag7));
    u64 gv;
    {
        const u64* g = (const u64*)(myws + WS_GLB) + (par * NREP + idx) * 16 + (lane & 7);
        const u64 t0 = __builtin_amdgcn_s_memrealtime();
        for (unsigned it = 1;; ++it) {
            gv = __hip_atomic_load(g, __ATOMIC_RELAXED, __HIP_MEMORY_SCOPE_SYSTEM);
            if (__all((unsigned)(gv >> 32) == call)) break;
            if ((it & 63u) == 0u && __builtin_amdgcn_readfirstlane((int)give_up(tmo, call, t0))) return;
            __builtin_amdgcn_s_sleep(1);
        }
    }
    STAMP(3);
    const float amax = __uint_as_float(wave_umax((unsigned)gv));
    const float inv = amax > 0.f ? 127.0f / amax : 0.f;
#define SEND(c) do { u32x4 ov; \
        const unsigned e0 = __float_as_uint(fmaf(ex[c][0], inv, 12582912.0f)), e1 = __float_as_uint(fmaf(ex[c][1], inv, 12582912.0f)); \
        const unsigned e7 = ((__builtin_amdgcn_perm(e1, e0, 0x0c0c0400u) & 0x7f7fu) << 7) | tag7; \
        _Pragma("unroll") for (int f = 0; f < 4; ++f) { \
            const unsigned top = __umul24((e7 >> (4 * f)) & 0x780u, 0x204081u);        \
            ov[f] = (quant4(acc[c][f][0], acc[c][f][1], acc[c][f][2], acc[c][f][3], inv, slo, shi) & 0x7f7f7f7fu) | (top & 0x80808080u); } \
          \
        if ((c) >= 1 && (c) <= 3) WAITV(0); \
        if ((c) == 1 || (c) == 2) { \
            __builtin_amdgcn_raw_buffer_store_b128(ov, rd, lane < 32 ? coff[c] : OOB, slot * 8192, 17); WAITV(0); \
            __builtin_amdgcn_raw_buffer_store_b128(ov, rd, lane < 32 ? OOB : coff[c], slot * 8192, 17); \
        } else __builtin_amdgcn_raw_buffer_store_b128(ov, rd, coff[c], slot * 8192, 17); } while (0)
    SEND(0);
    if (nch > 7) SEND(7);
    if (nch > 6) SEND(6);
    if (nch > 5) SEND(5);
    if (nch > 4) SEND(4);
    if (nch > 3) SEND(3);
    if (nch > 2) SEND(2);
    if (nch > 1) SEND(1);
    STAMP(4);
}

extern "C" void kernel_launch(int rank, int nranks, ncclComm_t comm, hipStream_t stream,
                              void* const* d_in, const int* in_sizes, int n_in,
                              void* d_out, int out_size, const int* in_dtypes, int out_dtype,
                              void* const* d_out_all, void* const* d_ws_all, size_t ws_bytes) {
    if (nranks != NR || n_in < 2 || in_sizes[0] != ML * KD || in_sizes[1] != KD * ND || out_size != NR * ML * NSH || ws_bytes < WS_NEED || (unsigned)rank >= (unsigned)NR) {
        fprintf(stderr, "kernel_launch: unexpected problem shape\n");
        return;
    }
    Params p;
    p.x = (const unsigned short*)d_in[0];
    p.w = (const unsigned short*)d_in[1];
    p.out = (char*)d_out;
    for (int d = 0; d < NR; ++d) p.ws[d] = (char*)d_ws_all[(rank + d) & 7];
    p.rank = rank; p.pad = 0;
    hipLaunchKernelGGL(fused_gemm_a2a, dim3(NWG), dim3(512), 0, stream, p);
    HIP_CHECK(hipGetLastError());
}
```

```cpp
#include <hip/hip_runtime.h>
#include <rccl/rccl.h>
#include <cstdio>

#define HIP_CHECK(expr) do { hipError_t _e = (expr); if (_e != hipSuccess) { \
    fprintf(stderr, "HIP error %s at %s:%d\n", hipGetErrorString(_e), __FILE__, __LINE__); \
    return; } } while (0)

#define LAS __attribute__((address_space(3)))
typedef short v4s __attribute__((ext_vector_type(4)));
typedef short v8s __attribute__((ext_vector_type(8)));
typedef float f32x4 __attribute__((ext_vector_type(4)));
typedef unsigned u32x4 __attribute__((ext_vector_type(4)));
typedef unsigned long long u64;

#define TMIN 78
#define TEND 44
#define TPH 10
#define TSLK 40

namespace {
constexpr int NR = 8;
constexpr int KD = 4096, ND = 16384, ML = 512, NSH = ND / NR;
constexpr int BM = 256, BN = 128, BK = 64, NT = KD / BK;
constexpr int NWG = 256;
constexpr int KROT = 8;
constexpr int LDS_B = 0, LDS_A = 65536, LDS_X = 131072, LDS_LUT = LDS_X + 64, LDS_TOT = 163840;
constexpr int NREP = 32;
constexpr int WS_CNT = 0;
constexpr int WS_TMO = 4096;
constexpr int WS_LOC = 8192;
constexpr int WS_GLB = 16384;
constexpr int WS_STAMP = 65536;
constexpr int WS_MSK = 1 << 20;
constexpr int WS_DAT = 3 << 20;
constexpr int MSK_PAR = 1 << 20, DAT_BYTES = 8 << 20;
constexpr size_t WS_NEED = (size_t)WS_DAT + (size_t)DAT_BYTES;
constexpr u64 DEADLINE = 150000000ull;
constexpr int OOB = 0x40000000;

struct Params {
    const unsigned short* x; const unsigned short* w;
    int rank, pad;
    char* out; char* ws[NR];
};
}

#define SB0 __builtin_amdgcn_sched_barrier(0)
#define WAITV(n) asm volatile("s_waitcnt vmcnt(%0)" :: "n"(n) : "memory")
#define WAITL0 do { asm volatile("" ::: "memory"); __builtin_amdgcn_s_waitcnt(0xC07F); asm volatile("" ::: "memory"); } while (0)
#define BAR do { asm volatile("" ::: "memory"); __builtin_amdgcn_s_barrier(); asm volatile("" ::: "memory"); } while (0)
#define STAMP(i) do { } while (0)

__device__ __forceinline__ v8s cat(v4s a, v4s b) { return __builtin_shufflevector(a, b, 0, 1, 2, 3, 4, 5, 6, 7); }
__device__ __forceinline__ int imax3(int a, float b, float c) { return max(max(a, __float_as_int(b)), __float_as_int(c)); }
__device__ __forceinline__ float relu(float a) { return __int_as_float(max(__float_as_int(a), 0)); }
__device__ __forceinline__ unsigned wave_umax(unsigned v) {
    v = max(v, (unsigned)__builtin_amdgcn_update_dpp(0, (int)v, 0xB1, 0xf, 0xf, true));
    v = max(v, (unsigned)__builtin_amdgcn_update_dpp(0, (int)v, 0x4E, 0xf, 0xf, true));
    v = max(v, (unsigned)__builtin_amdgcn_update_dpp(0, (int)v, 0x141, 0xf, 0xf, true));
    v = max(v, (unsigned)__builtin_amdgcn_update_dpp(0, (int)v, 0x140, 0xf, 0xf, true));
    const unsigned a = __builtin_amdgcn_readlane(v, 0), b = __builtin_amdgcn_readlane(v, 16), c = __builtin_amdgcn_readlane(v, 32), d = __builtin_amdgcn_readlane(v, 48);
    return max(max(a, b), max(c, d));
}
__device__ __forceinline__ bool give_up(unsigned* tmo, unsigned call, u64 t0) {
    if (__hip_atomic_load(tmo, __ATOMIC_RELAXED, __HIP_MEMORY_SCOPE_AGENT) == call || __builtin_amdgcn_s_memrealtime() - t0 > DEADLINE) {
        __hip_atomic_store(tmo, call, __ATOMIC_RELAXED, __HIP_MEMORY_SCOPE_AGENT);
        return true;
    }
    return false;
}
__device__ __forceinline__ unsigned quant4(float a, float b, float c, float d, float inv, unsigned slo, unsigned shi) {
    const unsigned q0 = __float_as_uint(fmaf(a, inv, 12582912.0f)), q1 = __float_as_uint(fmaf(b, inv, 12582912.0f));
    const unsigned q2 = __float_as_uint(fmaf(c, inv, 12582912.0f)), q3 = __float_as_uint(fmaf(d, inv, 12582912.0f));
    return __builtin_amdgcn_perm(q1, q0, slo) | __builtin_amdgcn_perm(q3, q2, shi);
}
__device__ __forceinline__ int chunk_off(int ng, int fq, int c) {
    const bool act = ng > 4 * c + fq;
    const u64 bal = __ballot(act);
    const int rank = __builtin_amdgcn_mbcnt_hi((unsigned)(bal >> 32), __builtin_amdgcn_mbcnt_lo((unsigned)bal, 0));
    return act ? c * 1024 + rank * 16 : OOB;
}
__device__ __forceinline__ void pool_scan(int n, int fr, int fq, int& base, int& ng) {
    const int n0 = __shfl(n, fr), n1 = __shfl(n, fr + 16), n2 = __shfl(n, fr + 32), n3 = __shfl(n, fr + 48);
    base = (fq > 0 ? n0 : 0) + (fq > 1 ? n1 : 0) + (fq > 2 ? n2 : 0);
    ng = (((n0 + n1) + (n2 + n3) + 17) * 3641) >> 16;
}
__device__ __forceinline__ unsigned top_bits(u32x4 v) {
    const unsigned lo = __builtin_amdgcn_udot4(v[1] & 0x80808080u, 0x80402010u, __builtin_amdgcn_udot4(v[0] & 0x80808080u, 0x08040201u, 0u, false), false);
    const unsigned hi = __builtin_amdgcn_udot4(v[3] & 0x80808080u, 0x80402010u, __builtin_amdgcn_udot4(v[2] & 0x80808080u, 0x08040201u, 0u, false), false);
    return (lo >> 7) | (hi << 1);
}
__device__ __forceinline__ unsigned call_tag(unsigned call) { const unsigned t = call % 3u + 1u; return ((t & 1u) << 7) | ((t & 2u) << 14); }

__global__ __launch_bounds__(512, 2) void fused_gemm_a2a(Params p) {
    __shared__ __attribute__((aligned(1024))) char smem[LDS_TOT];
    LAS char* lds = (LAS char*)smem;
    volatile LAS unsigned* xs = (volatile LAS unsigned*)(lds + LDS_X);
    const int tid = threadIdx.x, lane = tid & 63;
    const int wid = __builtin_amdgcn_readfirstlane(tid >> 6);
    const int b = blockIdx.x, dsel = b & 7, idx = b >> 3;
    const int rb = idx & 1, cb = idx >> 1;
    const int m0 = rb * BM;
    const int n0 = ((p.rank + dsel) & 7) * NSH + cb * BN;
    auto rx = __builtin_amdgcn_make_buffer_rsrc((void*)p.x, 0, ML * KD * 2, 0x00020000);
    auto rw = __builtin_amdgcn_make_buffer_rsrc((void*)p.w, 0, KD * ND * 2, 0x00020000);
    const int voA = (m0 + wid * 8 + (lane >> 3)) * (KD * 2) + (((lane & 7) ^ ((wid & 1) * 4 + (lane >> 4))) * 16);
    const int voB = (wid * 4 + (lane >> 4)) * (ND * 2) + n0 * 2 + (((lane & 15) ^ (((wid & 2) << 2) | (lane >> 4))) * 16);
    {
        const int kt = (dsel * KROT) & (NT - 1);
#pragma unroll
        for (int j = 0; j < 2; ++j)
            __builtin_amdgcn_raw_ptr_buffer_load_lds(rw, (LAS void*)(lds + LDS_B + wid * 1024 + j * 8192), 16, voB, kt * (BK * ND * 2) + j * (32 * ND * 2), 0, 0);
#pragma unroll
        for (int j = 0; j < 4; ++j)
            __builtin_amdgcn_raw_ptr_buffer_load_lds(rx, (LAS void*)(lds + LDS_A + (j >> 1) * 49152 + wid * 1024 + (j & 1) * 8192), 16, voA, kt * (BK * 2) + j * (64 * KD * 2), 0, 0);
    }
    SB0;
    char* myws = p.ws[0];
    unsigned* tmo = (unsigned*)(myws + WS_TMO);
    unsigned call;
#define GET_CALL() do { call = *(const volatile __attribute__((address_space(4))) unsigned*)(myws + WS_CNT) + 1u; asm volatile("" : "+s"(call)); } while (0)
    const int fr = lane & 15, fq = lane >> 4;
    STAMP(0);

    if (__builtin_expect(wid >= 4, 1)) {
        const int lw = wid - 4;
        {
            const int vA = voA - 32 * (KD * 2), vB = voB - 16 * (ND * 2);
            const int ksw = (rb ^ cb) & 1;
            auto kt_of = [&](int t) { return ((t < 2 ? t : t ^ ksw) + dsel * KROT) & (NT - 1); };
            auto issueW = [&](int t) {
                const int so = kt_of(t) * (BK * ND * 2);
#pragma unroll
                for (int j = 0; j < 4; ++j)
                    __builtin_amdgcn_raw_ptr_buffer_load_lds(rw, (LAS void*)(lds + LDS_B + (t & 3) * 16384 + lw * 1024 + j * 4096), 16, vB, so + j * (16 * ND * 2), 0, 0);
            };
            auto issueX = [&](int t, int so) {
                const int kt = kt_of(t);
#pragma unroll
                for (int j = 0; j < 8; ++j)
                    __builtin_amdgcn_raw_ptr_buffer_load_lds(rx, (LAS void*)(lds + LDS_A + (j >> 2) * 49152 + so + lw * 1024 + (j & 3) * 4096), 16, vA, kt * (BK * 2) + j * (32 * KD * 2), 0, 0);
            };
            issueW(1); issueW(2); issueX(1, 16384); issueW(3); issueX(2, 32768);
            GET_CALL();
            WAITV(28); BAR;
            WAITV(24); BAR;
            unsigned tnext = (unsigned)__builtin_amdgcn_s_memrealtime() - dsel * TPH;
            int so = 0;
            for (int t = 0; t < NT - 4; ++t) {
                if (TMIN && lw == 3 && t < TEND) {
                    unsigned now;
                    while ((int)((now = (unsigned)__builtin_amdgcn_s_memrealtime()) - tnext) < 0) __builtin_amdgcn_s_sleep(0);
                    now -= TSLK;
                    tnext = ((int)(now - tnext) > 0 ? now : tnext) + TMIN;
                }
                WAITV(12);
                BAR;
                issueW(t + 4); issueX(t + 3, so);
                so = (so == 32768) ? 0 : so + 16384;
            }
            WAITV(12); BAR;
            issueX(NT - 1, so);
            WAITV(8); BAR;
            {
                auto rm = __builtin_amdgcn_make_buffer_rsrc((void*)(myws + WS_MSK), 0, 2 * MSK_PAR, 0x00020000);
                WAITV(0);
                __builtin_amdgcn_raw_buffer_store_b128((u32x4){0u, 0u, 0u, 0u}, rm, lane * 16, ((call & 1) ^ 1) * MSK_PAR + (b * 4 + lw) * 1024, 17);
            }
            BAR;
            BAR;
            WAITV(0);
        }
        const int par = call & 1;
        if (lw == 2) {
            const int nib = lane & 15, al = lane >> 4;
            unsigned sel = 0;
#pragma unroll
            for (int r = 0; r < 4; ++r) sel |= (((nib >> r) & 1) ? (unsigned)(al + __builtin_popcount(nib & ((1 << r) - 1))) : 0x0cu) << (8 * r);
            *(LAS unsigned*)(lds + LDS_LUT + lane * 4) = sel;
            if (lane < 4) xs[4 + lane] = 0u;
            WAITL0;
        }
        BAR;
        if (lw == 0 && lane == 0) {
            const unsigned mx = max(max(xs[0], xs[1]), max(xs[2], xs[3]));
            __hip_atomic_store((u64*)(myws + WS_LOC) + par * NWG + b, ((u64)call << 32) | mx, __ATOMIC_RELAXED, __HIP_MEMORY_SCOPE_AGENT);
        }
        STAMP(1);
        if (lw == 1 && b == 0) {
            char* base = p.ws[0];
#pragma unroll
            for (int r = 1; r < NR; ++r) base = ((lane & 7) == r) ? p.ws[r] : base;
            typedef __attribute__((address_space(1))) u64 gu64;
            gu64* dst = (gu64*)(base + WS_GLB) + (par * NREP + (lane >> 3)) * 16 + p.rank;
            asm volatile("" : "+v"(dst));
            const u64* tb = (const u64*)(myws + WS_LOC) + par * NWG + lane;
            const u64 t0 = __builtin_amdgcn_s_memrealtime();
            unsigned mx; bool dead = false;
            for (unsigned it = 1;; ++it) {
                bool ok = true; mx = 0;
#pragma unroll
                for (int i = 0; i < 4; ++i) { const u64 v = __hip_atomic_load(tb + i * 64, __ATOMIC_RELAXED, __HIP_MEMORY_SCOPE_AGENT); ok &= (unsigned)(v >> 32) == call; mx = max(mx, (unsigned)v); }
                if (__all(ok)) break;
                if ((it & 63u) == 0u && __builtin_amdgcn_readfirstlane((int)give_up(tmo, call, t0))) { dead = true; break; }
                __builtin_amdgcn_s_sleep(1);
            }
            mx = wave_umax(mx);
            if (!dead) {
#pragma unroll
                for (int i = 0; i < NREP / 8; ++i) __hip_atomic_store(dst + i * 128, ((u64)call << 32) | mx, __ATOMIC_RELAXED, __HIP_MEMORY_SCOPE_SYSTEM);
            }
            if (lane == 0) __hip_atomic_store((unsigned*)(myws + WS_CNT), call, __ATOMIC_RELAXED, __HIP_MEMORY_SCOPE_AGENT);
            STAMP(6);
        }
        const int slot = b * 4 + lw;
        const u64 t0 = __builtin_amdgcn_s_memrealtime();
        bool ok = true;
        u32x4 kept;
        {
            auto rm = __builtin_amdgcn_make_buffer_rsrc((void*)(myws + WS_MSK), 0, 2 * MSK_PAR, 0x00020000);
            for (unsigned it = 1;; ++it) {
                kept = __builtin_amdgcn_raw_buffer_load_b128(rm, lane * 16, par * MSK_PAR + slot * 1024, 17);
                if (__all((int)kept[3] < 0)) break;
                if ((it & 63u) == 0u && __builtin_amdgcn_readfirstlane((int)give_up(tmo, call, t0))) { ok = false; break; }
                __builtin_amdgcn_s_sleep(16);
                asm volatile("" ::: "memory");
            }
        }
        STAMP(2);
        const int sbase = lw * 32768 + fr * 4;
        int wa[32]; unsigned sel[32]; int n, ng;
        pool_scan(__builtin_popcount(kept[0]) + __builtin_popcount(kept[1]) + __builtin_popcount(kept[2]) + __builtin_popcount(kept[3]), fr, fq, n, ng);
#pragma unroll
        for (int w = 0; w < 4; ++w) {
#pragma unroll
            for (int j = 0; j < 8; ++j) {
                const int sh = 4 * (j ^ (fq & 1));
                const int pos = n + __builtin_popcount(__builtin_amdgcn_ubfe(kept[w], 0, sh));
                const int nib = __builtin_amdgcn_ubfe(kept[w], sh, 4);
                wa[w * 8 + j] = sbase + (pos >> 2) * 64;
                sel[w * 8 + j] = *(LAS unsigned*)(lds + LDS_LUT + (((pos & 3) << 4) | nib) * 4);
            }
            n += __builtin_popcount(kept[w]);
        }
        int coff[8];
#pragma unroll
        for (int c = 0; c < 8; ++c) coff[c] = chunk_off(ng, fq, c);
        const int nch = (int)((wave_umax((unsigned)ng) + 3u) >> 2);
        const int xbase = sbase + (4 * ng + (fq >> 1)) * 64 + (fq & 1) * 2;
#pragma unroll
        for (int i = 0; i < 32; ++i) { asm volatile("" : "+v"(wa[i])); asm volatile("" : "+v"(sel[i])); }
        const int src = (p.rank - dsel) & 7;
        int o = (src * ML + m0 + (lw >> 1) * 128 + fr) * NSH + cb * BN + (lw & 1) * 64 + fq * 16;
        asm volatile("" : "+v"(o));
        auto ro = __builtin_amdgcn_make_buffer_rsrc((void*)p.out, 0, NR * ML * NSH, 0x00020000);
        const unsigned tag = call_tag(call);
        while (xs[4 + lw] == 0u) __builtin_amdgcn_s_sleep(1);
        asm volatile("" ::: "memory");
        STAMP(3);
        {
            auto rd = __builtin_amdgcn_make_buffer_rsrc((void*)(myws + WS_DAT), 0, DAT_BYTES, 0x00020000);
            for (unsigned it = 1; ok; ++it) {
                const u32x4 v = __builtin_amdgcn_raw_buffer_load_b128(rd, lane == 0 ? 0 : OOB, slot * 8192, 17);
                if (__builtin_amdgcn_readfirstlane((top_bits(v) & 0x8080u) == tag)) break;
                if ((it & 63u) == 0u && __builtin_amdgcn_readfirstlane((int)give_up(tmo, call, t0))) ok = false;
                __builtin_amdgcn_s_sleep(8);
                asm volatile("" ::: "memory");
            }
            STAMP(7);
            for (unsigned it = 1; ok; ++it) {
                u32x4 v[8];
#pragma unroll
                for (int c = 0; c < 8; ++c) if (c < 4 || c < nch) v[c] = __builtin_amdgcn_raw_buffer_load_b128(rd, coff[c], slot * 8192, 17);
                int left = OOB;
#pragma unroll
                for (int c = 0; c < 8; ++c) if (c < 4 || c < nch) {
                    const unsigned h = top_bits(v[c]);
                    if ((h & 0x8080u) == tag) {
#pragma unroll
                        for (int i = 0; i < 4; ++i) *(LAS unsigned*)(lds + sbase + fq * 256 + (c * 16 + i) * 64) = v[c][i] & 0x7f7f7f7fu;
                        *(LAS unsigned short*)(lds + xbase + c * 128) = (unsigned short)(h & 0x7f7fu);
                        __builtin_amdgcn_raw_buffer_store_b128((u32x4){0u, 0u, 0u, 0u}, rd, coff[c], slot * 8192, 17);
                        coff[c] = OOB;
                    }
                    left = min(left, coff[c]);
                }
                if (__all(left == OOB)) break;
                if ((it & 63u) == 0u && __builtin_amdgcn_readfirstlane((int)give_up(tmo, call, t0))) ok = false;
                __builtin_amdgcn_s_sleep(0);
                asm volatile("" ::: "memory");
            }
        }
        STAMP(4);
#pragma unroll
        for (int m = 0; m < 8; ++m) {
            u32x4 ov;
#pragma unroll
            for (int j = 0; j < 4; ++j) {
                const unsigned lo = *(LAS unsigned*)(lds + wa[m * 4 + j]), hi = *(LAS unsigned*)(lds + wa[m * 4 + j] + 64);
                ov[j] = __builtin_amdgcn_perm(hi, lo, sel[m * 4 + j]);
            }
            if (!ok) ov = (u32x4){0x80808080u, 0x80808080u, 0x80808080u, 0x80808080u};
            __builtin_amdgcn_raw_buffer_store_b128(ov, ro, o, m * 16 * NSH, 16);
        }
        STAMP(5);
        return;
    }
    const int wm = wid >> 1, wn = wid & 1;
    const int aA0 = LDS_A + wm * 49152 + fr * 128 + ((fq ^ (fr >> 1)) * 16);
    const int aA1 = aA0 ^ 64;
    int aB[4];
    {
        const int q = fr >> 2, pp = lane & 3;
        const int ch = (wn * 8 + 2 * pp) ^ (((fq & 1) << 3) | q);
        const int a0 = LDS_B + (8 * fq + q) * 256 + ch * 16 + (pp & 1) * 8;
#pragma unroll
        for (int f = 0; f < 4; ++f) aB[f] = a0 ^ (8 * f);
    }
    f32x4 acc[8][4];
#pragma unroll
    for (int m = 0; m < 8; ++m)
#pragma unroll
        for (int f = 0; f < 4; ++f) { acc[m][f] = (f32x4){0.f, 0.f, 0.f, 0.f}; asm volatile("" : "+v"(acc[m][f])); }
    v8s A[4][2];
    v4s B0[4][2][2], B1[4][2][2];
#define RA(m, ks, so) A[(m) & 3][ks] = *(LAS v8s*)(lds + ((ks) ? aA1 : aA0) + (so) + (m) * 2048)
#define RB(Bv, i, sb) Bv[(i) >> 2][((i) >> 1) & 1][(i) & 1] = __builtin_amdgcn_ds_read_tr16_b64_v4i16((LAS v4s*)(lds + aB[(i) >> 2] + (sb) * 16384 + (((i) >> 1) & 1) * 8192 + ((i) & 1) * 1024))
#define MF(m, f, ks, Bv) asm volatile("v_mfma_f32_16x16x32_bf16 %0, %1, %2, %0" : "+v"(acc[m][f]) : "v"(cat(Bv[f][ks][0], Bv[f][ks][1])), "v"(A[(m) & 3][ks]) : "memory")
#define MF2(m, f, Bv) asm volatile("v_mfma_f32_16x16x32_bf16 %0, %1, %2, %0\n\tv_mfma_f32_16x16x32_bf16 %0, %3, %4, %0" : "+v"(acc[m][f]) : "v"(cat(Bv[f][0][0], Bv[f][0][1])), "v"(A[(m) & 3][0]), "v"(cat(Bv[f][1][0], Bv[f][1][1])), "v"(A[(m) & 3][1]) : "memory")
    GET_CALL();
    WAITV(0);
    BAR;
    STAMP(1);
    WAITL0; SB0;
#pragma unroll
    for (int i = 0; i < 12; ++i) { RB(B0, i, 0); SB0; }
    RA(0, 0, 0); SB0; RA(0, 1, 0); SB0; RB(B0, 12, 0); SB0; RB(B0, 13, 0); SB0; RA(1, 0, 0); SB0; RA(1, 1, 0); SB0; RB(B0, 14, 0); SB0; RB(B0, 15, 0); SB0; RA(2, 0, 0); SB0; RA(2, 1, 0); SB0;
    BAR;
    int so = 0;
#define CH(m, f, Bc, R) MF(m, f, 0, Bc); R; SB0; MF(m, f, 1, Bc); SB0;
#define GA(m, Bc, Bn, sb, ma, soa) CH(m, 0, Bc, RA(ma, 0, soa)) CH(m, 1, Bc, RA(ma, 1, soa)) CH(m, 2, Bc, RB(Bn, 2 * (m), sb)) CH(m, 3, Bc, RB(Bn, 2 * (m) + 1, sb))
#define GB(m, Bc, Bn, sb, ma, soa) CH(m, 0, Bc, RB(Bn, 2 * (m), sb)) CH(m, 1, Bc, RB(Bn, 2 * (m) + 1, sb)) CH(m, 2, Bc, RA(ma, 0, soa)) CH(m, 3, Bc, RA(ma, 1, soa))
#define KTILE(Bc, Bn, sb) { \
        const int sn = (so == 32768) ? 0 : so + 16384; \
        SB0; \
        GA(0, Bc, Bn, sb, 3, so) GA(1, Bc, Bn, sb, 4, so) GA(2, Bc, Bn, sb, 5, so) GA(3, Bc, Bn, sb, 6, so) GA(4, Bc, Bn, sb, 7, so) \
        CH(5, 0, Bc, RB(Bn, 10, sb)) CH(5, 1, Bc, RB(Bn, 11, sb)) MF2(5, 2, Bc); SB0; \
        asm volatile("" ::: "memory"); __builtin_amdgcn_s_waitcnt(0xC47F);   BAR; SB0; \
        RA(0, 0, sn); SB0; CH(5, 3, Bc, RA(0, 1, sn)) \
        GB(6, Bc, Bn, sb, 1, sn) GB(7, Bc, Bn, sb, 2, sn) \
        so = sn; }
    KTILE(B0, B1, 1)
    KTILE(B1, B0, 2)
    KTILE(B0, B1, 3)
    for (int t = 3; t < NT - 1; t += 4) {
        KTILE(B1, B0, 0)
        KTILE(B0, B1, 1)
        KTILE(B1, B0, 2)
        KTILE(B0, B1, 3)
    }
    int mx = 0;
    unsigned sg[4] = {0u, 0u, 0u, 0u};
#define EPA(m, f) asm volatile("v_max3_i32 %0, %0, %2, %3\n\tv_alignbit_b32 %1, %1, %2, 31\n\tv_alignbit_b32 %1, %1, %3, 31" : "+v"(mx), "+v"(sg[(m) >> 1]) : "v"(acc[m][f][0]), "v"(acc[m][f][1]))
#define EPB(m, f) asm volatile("v_max3_i32 %0, %0, %2, %3\n\tv_alignbit_b32 %1, %1, %2, 31\n\tv_alignbit_b32 %1, %1, %3, 31" : "+v"(mx), "+v"(sg[(m) >> 1]) : "v"(acc[m][f][2]), "v"(acc[m][f][3]))
#define CE(m, f) MF(m, f, 0, B1); EPA((m) - 1, f); SB0; MF(m, f, 1, B1); EPB((m) - 1, f); SB0;
#define GE(m) RA((m) + 3, 0, so); RA((m) + 3, 1, so); SB0; CE(m, 0) CE(m, 1) CE(m, 2) CE(m, 3)
    SB0;
    CH(0, 0, B1, RA(3, 0, so)) CH(0, 1, B1, RA(3, 1, so)) MF2(0, 2, B1); MF2(0, 3, B1); SB0;
    GE(1) GE(2) GE(3) GE(4)
    CE(5, 0) CE(5, 1)
    WAITL0; BAR; SB0;
    CE(5, 2) CE(5, 3) CE(6, 0) CE(6, 1) CE(6, 2) CE(6, 3) CE(7, 0) CE(7, 1) CE(7, 2) CE(7, 3)
    asm volatile("s_nop 7\n\ts_nop 7" ::: "memory");
    SB0;
    STAMP(2);
    {
#pragma unroll
        for (int f = 0; f < 4; ++f) mx = imax3(imax3(mx, acc[7][f][0], acc[7][f][1]), acc[7][f][2], acc[7][f][3]);
        const unsigned wmx = wave_umax((unsigned)mx);
        if (lane == 0) xs[wid] = wmx;
        WAITL0;
        BAR;
    }
    const int par = call & 1;
    const int slot = b * 4 + wid;
    char* pws = p.ws[dsel];
    acc[7][3][3] = relu(acc[7][3][3]);
#pragma unroll
    for (int i = 0; i < 16; ++i) sg[3] = __builtin_amdgcn_alignbit(sg[3], __float_as_uint(acc[7][i >> 2][i & 3]), 31);
    u32x4 kept;
#pragma unroll
    for (int w = 0; w < 4; ++w) kept[w] = ~__builtin_bitreverse32(sg[w]);
    auto rm = __builtin_amdgcn_make_buffer_rsrc((void*)(pws + WS_MSK), 0, 2 * MSK_PAR, 0x00020000);
#define SEND_MAP(k) do { if (wid == (k)) __builtin_amdgcn_raw_buffer_store_b128(kept, rm, lane * 16, par * MSK_PAR + slot * 1024, 17); } while (0)
    int base, ng;
    pool_scan(__builtin_popcount(kept[0]) + __builtin_popcount(kept[1]) + __builtin_popcount(kept[2]) + __builtin_popcount(kept[3]), fr, fq, base, ng);
    const int nch = (int)((wave_umax((unsigned)ng) + 3u) >> 2);
    float ex[8][2];
    {
        int a = wid * 32768 + fr * 4 + (384 - base) * 64;
#pragma unroll
        for (int v = 0; v < 128; ++v) {
            if (v % 40 == 0) SEND_MAP(v / 40);
            const float y = acc[v >> 4][(v >> 2) & 3][v & 3];
            *(LAS float*)(lds + a + (127 - v) * 64) = y;
            int t;
            if (v < 127) asm("v_lshrrev_b32 %1, 31, %2\n\tv_lshl_add_u32 %0, %1, 6, %0" : "+v"(a), "=&v"(t) : "v"(y));
        }
        asm volatile("" ::: "memory");
        SB0;
        const int rbase = wid * 32768 + fr * 4 + (48 - 16 * fq) * 64;
        int re = rbase + 64 - (fq & 1) * 64, ro = rbase + (fq & 1) * 64;
        asm volatile("" : "+v"(re), "+v"(ro));
#pragma unroll
        for (int c = 0; c < 8; ++c) {
            ex[c][0] = 0.f; ex[c][1] = 0.f;
            if (c < 4 || c < nch) {
#pragma unroll
                for (int i = 0; i < 16; ++i) acc[c][i >> 2][i & 3] = *(LAS float*)(lds + ((i & 1) ? ro : re) + (462 + (i & 1) - 64 * c - i) * 64);
                const int xa = wid * 32768 + fr * 4 + (510 - min(16 * ng + 2 * (4 * c + fq), 510)) * 64;
                ex[c][0] = *(LAS float*)(lds + xa + 64); ex[c][1] = *(LAS float*)(lds + xa);
            }
        }
    }
    int coff[8];
#pragma unroll
    for (int c = 0; c < 8; ++c) coff[c] = chunk_off(ng, fq, c);
    unsigned slo = (fq & 1) ? 0x0c0c0004u : 0x0c0c0400u, shi = (fq & 1) ? 0x00040c0cu : 0x04000c0cu;
    asm volatile("" : "+v"(slo), "+v"(shi));
#pragma unroll
    for (int m = 0; m < 8; ++m) {
        asm volatile("" : "+v"(coff[m]), "+v"(ex[m][0]), "+v"(ex[m][1]));
#pragma unroll
        for (int f = 0; f < 4; ++f) asm volatile("" : "+v"(acc[m][f]));
    }
    WAITL0;
    if (lane == 0) xs[4 + wid] = 1u;
    auto rd = __builtin_amdgcn_make_buffer_rsrc((void*)(pws + WS_DAT), 0, DAT_BYTES, 0x00020000);
    STAMP(6);
    unsigned tag7 = call_tag(call) << 7;
    asm volatile("" : "+s"(tag7));
    u64 gv;
    {
        const u64* g = (const u64*)(myws + WS_GLB) + (par * NREP + idx) * 16 + (lane & 7);
        const u64 t0 = __builtin_amdgcn_s_memrealtime();
        for (unsigned it = 1;; ++it) {
            gv = __hip_atomic_load(g, __ATOMIC_RELAXED, __HIP_MEMORY_SCOPE_SYSTEM);
            if (__all((unsigned)(gv >> 32) == call)) break;
            if ((it & 63u) == 0u && __builtin_amdgcn_readfirstlane((int)give_up(tmo, call, t0))) return;
            __builtin_amdgcn_s_sleep(1);
        }
    }
    STAMP(3);
    const float amax = __uint_as_float(wave_umax((unsigned)gv));
    const float inv = amax > 0.f ? 127.0f / amax : 0.f;
#define SEND(c) do { u32x4 ov; \
        const unsigned e0 = __float_as_uint(fmaf(ex[c][0], inv, 12582912.0f)), e1 = __float_as_uint(fmaf(ex[c][1], inv, 12582912.0f)); \
        const unsigned e7 = ((__builtin_amdgcn_perm(e1, e0, 0x0c0c0400u) & 0x7f7fu) << 7) | tag7; \
        _Pragma("unroll") for (int f = 0; f < 4; ++f) { \
            const unsigned top = __umul24((e7 >> (4 * f)) & 0x780u, 0x204081u);        \
            ov[f] = (quant4(acc[c][f][0], acc[c][f][1], acc[c][f][2], acc[c][f][3], inv, slo, shi) & 0x7f7f7f7fu) | (top & 0x80808080u); } \
          \
        if ((c) >= 1 && (c) <= 3) WAITV(0); \
        if ((c) == 1 || (c) == 2) { \
            __builtin_amdgcn_raw_buffer_store_b128(ov, rd, lane < 32 ? coff[c] : OOB, slot * 8192, 17); WAITV(0); \
            __builtin_amdgcn_raw_buffer_store_b128(ov, rd, lane < 32 ? OOB : coff[c], slot * 8192, 17); \
        } else __builtin_amdgcn_raw_buffer_store_b128(ov, rd, coff[c], slot * 8192, 17); } while (0)
    SEND(0);
    if (nch > 7) SEND(7);
    if (nch > 6) SEND(6);
    if (nch > 5) SEND(5);
    if (nch > 4) SEND(4);
    if (nch > 3) SEND(3);
    if (nch > 2) SEND(2);
    if (nch > 1) SEND(1);
    STAMP(4);
}

extern "C" void kernel_launch(int rank, int nranks, ncclComm_t comm, hipStream_t stream,
                              void* const* d_in, const int* in_sizes, int n_in,
                              void* d_out, int out_size, const int* in_dtypes, int out_dtype,
                              void* const* d_out_all, void* const* d_ws_all, size_t ws_bytes) {
    if (nranks != NR || n_in < 2 || in_sizes[0] != ML * KD || in_sizes[1] != KD * ND || out_size != NR * ML * NSH || ws_bytes < WS_NEED || (unsigned)rank >= (unsigned)NR) {
        fprintf(stderr, "kernel_launch: unexpected problem shape\n");
        return;
    }
    Params p;
    p.x = (const unsigned short*)d_in[0];
    p.w = (const unsigned short*)d_in[1];
    p.out = (char*)d_out;
    for (int d = 0; d < NR; ++d) p.ws[d] = (char*)d_ws_all[(rank + d) & 7];
    p.rank = rank; p.pad = 0;
    hipLaunchKernelGGL(fused_gemm_a2a, dim3(NWG), dim3(512), 0, stream, p);
    HIP_CHECK(hipGetLastError());
}
```
